```python
import numpy as np

kernel_name = "gpu_ag_gemm_m4096_k4096_n16384_bf16_q8"


ABSMAX_RTOL = 0.02

COMPUTE_DTYPE = "bfloat16"
EPILOGUE = "quant_int8"
IN_QUANT = None
OUT_QUANT = "int8"
IN_DTYPES = {"x": "bfloat16", "w_mat": "bfloat16"}
WORLD_SIZE = 8

def _to_bf16(x):
    u = np.ascontiguousarray(x, dtype=np.float32).view(np.uint32)
    return ((u + 0x7FFF + ((u >> 16) & 1)) & 0xFFFF0000).view(np.float32)

def setup_inputs():

    x = _to_bf16(np.random.randn(4096, 4096).astype(np.float32))
    w_mat = _to_bf16((np.random.randn(4096, 16384) / np.sqrt(4096)).astype(np.float32))
    return {"x": x, "w_mat": w_mat}

def _quant_dequant(y, amax):
    scale = amax / 127.0
    q = np.clip(np.round(y / scale), -127.0, 127.0)
    return (q * scale).astype(np.float32)
def _epilogue(y):
    return _quant_dequant(y, np.abs(y).max())

def reference(x, w_mat):
    y = (x @ w_mat).astype(np.float32)
    return _epilogue(y)
```

```cpp
#include <hip/hip_runtime.h>
#include <rccl/rccl.h>
#include <cstdio>

#define HIP_CHECK(expr) do { hipError_t _e = (expr); if (_e != hipSuccess) { \
    fprintf(stderr, "HIP error %s at %s:%d\n", hipGetErrorString(_e), __FILE__, __LINE__); \
    return; } } while (0)

namespace {

#define LAS __attribute__((address_space(3)))
typedef short s16x4 __attribute__((ext_vector_type(4)));
typedef short s16x8 __attribute__((ext_vector_type(8)));
typedef float f32x4 __attribute__((ext_vector_type(4)));
typedef unsigned u32x4 __attribute__((ext_vector_type(4)));
typedef unsigned u32x2 __attribute__((ext_vector_type(2)));

constexpr int NR = 8;
constexpr int MS = 512;
constexpr int KD = 4096;
constexpr int NL = 2048;
constexpr int NWG = 256;
constexpr int BK = 32, NSTEP = KD / BK;
constexpr int NSTAGE = 6;
constexpr int A_STAGE = 256 * BK * 2;
constexpr int B_STAGE = BK * 128 * 2;
constexpr int STAGE = A_STAGE + B_STAGE;
constexpr int RING_BYTES = NSTAGE * STAGE;
constexpr int LDS_BYTES = RING_BYTES + 256;
constexpr int NRED = 2;

constexpr size_t WS_XS = 0;
constexpr size_t WS_FLAG = (size_t)MS * KD * 2;
constexpr size_t WS_LMAX = WS_FLAG + 65536;
constexpr size_t WS_GMAX = WS_LMAX + 65536;
constexpr size_t WS_CTR = WS_GMAX + 65536;
constexpr size_t WS_TMO = WS_CTR + 4096;
constexpr size_t WS_DBG = WS_TMO + 4096;
constexpr size_t WS_END = WS_DBG + 65536;
constexpr unsigned long long DEADLINE = 50000000ull;

struct Params {
    const char* x;
    const char* w;
    char* out;
    char* myws;
    char* ws[NR];
    int rank;
};

__device__ __forceinline__ u32x2 load8_sys(const char* p) {
    u32x2 v;
    asm volatile("global_load_dwordx2 %0, %1, off sc0 sc1\n\ts_waitcnt vmcnt(0)" : "=&v"(v) : "v"(p) : "memory");
    return v;
}
__device__ __forceinline__ void load16x2_sys(const char* p, u32x4& a, u32x4& b) {
    asm volatile("global_load_dwordx4 %0, %2, off sc0 sc1\n\tglobal_load_dwordx4 %1, %2, off offset:1024 sc0 sc1\n\ts_waitcnt vmcnt(0)"
                 : "=&v"(a), "=&v"(b) : "v"(p) : "memory");
}
__device__ __forceinline__ void store16_sys(char* p, u32x4 v) {
    asm volatile("global_store_dwordx4 %0, %1, off sc0 sc1\n\ts_nop 1" :: "v"(p), "v"(v) : "memory");
}
__device__ __forceinline__ unsigned umax(unsigned a, unsigned b) { return a > b ? a : b; }
__device__ __forceinline__ unsigned wave_umax(unsigned v) {
    v = umax(v, (unsigned)__builtin_amdgcn_update_dpp(0, (int)v, 0xB1, 0xf, 0xf, true));
    v = umax(v, (unsigned)__builtin_amdgcn_update_dpp(0, (int)v, 0x4E, 0xf, 0xf, true));
    v = umax(v, (unsigned)__builtin_amdgcn_update_dpp(0, (int)v, 0x141, 0xf, 0xf, true));
    v = umax(v, (unsigned)__builtin_amdgcn_update_dpp(0, (int)v, 0x140, 0xf, 0xf, true));
    const unsigned a = __builtin_amdgcn_readlane(v, 0), b = __builtin_amdgcn_readlane(v, 16), c = __builtin_amdgcn_readlane(v, 32), d = __builtin_amdgcn_readlane(v, 48);
    return umax(umax(a, b), umax(c, d));
}
__device__ __forceinline__ bool give_up(unsigned* tmo, unsigned call, unsigned long long t0, unsigned& polls) {
    if ((++polls & 15u) != 0u) return false;
    const unsigned t = __builtin_amdgcn_readfirstlane(__hip_atomic_load(tmo, __ATOMIC_RELAXED, __HIP_MEMORY_SCOPE_AGENT));
    if (t == call || __builtin_amdgcn_s_memrealtime() - t0 > DEADLINE) {
        __hip_atomic_store(tmo, call, __ATOMIC_RELAXED, __HIP_MEMORY_SCOPE_AGENT);
        return true;
    }
    return false;
}

#define STAMP(i) do { } while (0)

__global__ __launch_bounds__(512, 2) void ag_gemm_q8(Params P) {
    extern __shared__ __attribute__((aligned(16))) char smem[];
    LAS char* lds = (LAS char*)smem;
    const int tid = threadIdx.x, lane = tid & 63, wid = __builtin_amdgcn_readfirstlane(tid >> 6);
    const int bid = blockIdx.x;
    char* const myws = P.myws;
    unsigned* const tmo = (unsigned*)(myws + WS_TMO);
    unsigned* const ctr = (unsigned*)(myws + WS_CTR) + bid;
    STAMP(0);
#define AG_INV 1
#define AG_OST 0

    unsigned call;
    if (wid < 4) {
        const int kt = bid >> 2, q = bid & 3;
        const int r0 = tid >> 3, c = tid & 7;
        const char* src = P.x + (size_t)(q * 128 + r0) * (KD * 2) + kt * 128 + c * 16;
        u32x4 v[4];
#pragma unroll
        for (int i = 0; i < 4; ++i) v[i] = *(const u32x4*)(src + (size_t)i * 32 * (KD * 2));
        call = __hip_atomic_load(ctr, __ATOMIC_RELAXED, __HIP_MEMORY_SCOPE_AGENT) + 1u;
        char* flag = P.ws[tid & 7] + WS_FLAG + (size_t)(P.rank * NWG + bid) * 4;
        const int step = 2 * kt + (c >> 2), rt = (q & 1) * 128 + r0;
        char* dst = myws + WS_XS + (size_t)(step * 2 + (q >> 1)) * A_STAGE + (rt >> 4) * 1024 + ((c & 3) * 16 + (rt & 15)) * 16;
        asm volatile("" : "+v"(v[0]), "+v"(v[1]), "+v"(v[2]), "+v"(v[3]), "+v"(flag), "+v"(call));
#pragma unroll
        for (int i = 0; i < 4; ++i) store16_sys(dst + i * 2048, v[i]);
        asm volatile("s_waitcnt vmcnt(0)\n\ts_barrier" ::: "memory");
        if (tid < NR) asm volatile("global_store_dword %0, %1, off sc0 sc1" :: "v"(flag), "v"(call) : "memory");
    } else {
        call = __hip_atomic_load(ctr, __ATOMIC_RELAXED, __HIP_MEMORY_SCOPE_AGENT) + 1u;
        asm volatile("" : "+v"(call));
        if (AG_INV == 1 && wid == 4) asm volatile("buffer_inv sc0 sc1" ::: "memory");
        asm volatile("s_barrier" ::: "memory");
    }
    call = __builtin_amdgcn_readfirstlane(call);
    STAMP(1);

    const int srcr = bid & 7;
    const int tix = bid >> 3, half = tix & 1, n0 = (tix >> 1) * 128;

    int ready = 0;
    const char* const fp = myws + WS_FLAG + (size_t)srcr * (NWG * 4) + half * 8 + lane * 16;
    auto wait_ready = [&](int need) {
        const unsigned long long t0 = __builtin_amdgcn_s_memrealtime();
        unsigned polls = 0;
        for (;;) {
            const u32x2 f = load8_sys(fp);
            const unsigned long long miss = ~__ballot(f[0] >= call && f[1] >= call);
            ready = miss ? 2 * __builtin_ctzll(miss) : NSTEP;
            if (ready >= need) break;
            if (give_up(tmo, call, t0, polls)) { ready = NSTEP; break; }
        }
    };
    if (AG_INV == 2 && wid == 1) __builtin_amdgcn_fence(__ATOMIC_ACQUIRE, "");
    if (wid == 0) wait_ready(NSTAGE);
    asm volatile("s_waitcnt vmcnt(0) lgkmcnt(0)\n\ts_barrier" ::: "memory");
    STAMP(2);

    const unsigned lds0 = (unsigned)(size_t)lds;
    const char* gA = P.ws[srcr] + WS_XS + (size_t)half * A_STAGE + wid * 2048;
    const char* gB = P.w + n0 * 2;
    const unsigned voffA = lane * 16;
    unsigned voffB;
    {
        const int row = 8 * (wid >> 1) + ((lane >> 2) & 7), chh = 2 * (wid & 1) + (lane >> 5);
        const int ch = chh * 4 + ((lane & 3) ^ ((row >> 2) & 3) ^ (chh & 1));
        voffB = (unsigned)(row * (NL * 2) + ch * 16);
    }
    const unsigned ldsWA = lds0 + wid * 2048, ldsWB = lds0 + A_STAGE + wid * 1024;
    auto issue = [&](int slot_off) {
        unsigned keep;
        asm volatile("s_mov_b32 %0, m0\n\ts_mov_b32 m0, %1\n\ts_nop 3\n\t"
                     "global_load_lds_dwordx4 %3, %4\n\t"
                     "global_load_lds_dwordx4 %3, %4 offset:1024\n\t"
                     "s_mov_b32 m0, %2\n\ts_nop 0\n\t"
                     "global_load_lds_dwordx4 %5, %6\n\t"
                     "s_mov_b32 m0, %0"
                     : "=&s"(keep) : "s"(ldsWA + slot_off), "s"(ldsWB + slot_off), "v"(voffA), "s"(gA), "v"(voffB), "s"(gB) : "memory");
        gA += 2 * A_STAGE; gB += BK * NL * 2;
    };

    const int wm = wid >> 1, wc = wid & 1;
    const int g = lane >> 4, q4 = (lane & 15) >> 2, p4 = lane & 3;
    const unsigned rdA = wm * 4096 + lane * 16;
    const unsigned rdB0 = A_STAGE + 2048 * g + 512 * (2 * wc + (p4 >> 1)) + 64 * q4 + 32 * ((p4 ^ g) & 1);
    unsigned rdB[2][2];
#pragma unroll
    for (int u = 0; u < 2; ++u)
#pragma unroll
        for (int v = 0; v < 2; ++v) rdB[u][v] = rdB0 + 16 * (u ^ (p4 >> 1)) + 8 * (v ^ (p4 & 1));

    f32x4 acc[4][4];
#pragma unroll
    for (int i = 0; i < 4; ++i)
#pragma unroll
        for (int j = 0; j < 4; ++j) acc[i][j] = (f32x4){0.f, 0.f, 0.f, 0.f};

    auto compute = [&](int slot_off) {
        s16x8 xf[4], wf[4];
#pragma unroll
        for (int j = 0; j < 4; ++j) {
            const s16x4 lo = __builtin_amdgcn_ds_read_tr16_b64_v4i16((LAS s16x4*)(lds + slot_off + rdB[j >> 1][j & 1]));
            const s16x4 hi = __builtin_amdgcn_ds_read_tr16_b64_v4i16((LAS s16x4*)(lds + slot_off + rdB[(j >> 1) ^ 1][j & 1] + 256));
            wf[j] = __builtin_shufflevector(lo, hi, 0, 1, 2, 3, 4, 5, 6, 7);
        }
#pragma unroll
        for (int i = 0; i < 4; ++i) xf[i] = *(const LAS s16x8*)(lds + slot_off + rdA + i * 1024);
#pragma unroll
        for (int i = 0; i < 4; ++i)
#pragma unroll
            for (int j = 0; j < 4; ++j) acc[i][j] = __builtin_amdgcn_mfma_f32_16x16x32_bf16(wf[j], xf[i], acc[i][j], 0, 0, 0);
    };

#pragma unroll
    for (int s = 0; s < NSTAGE - 1; ++s) issue(s * STAGE);
    int rd = 0, wr = (NSTAGE - 1) * STAGE;
    for (int s = 0; s < NSTEP - (NSTAGE - 1); ++s) {
        if (wid == 0 && ready < s + NSTAGE) wait_ready(s + NSTAGE);
        asm volatile("s_waitcnt vmcnt(12) lgkmcnt(0)\n\ts_barrier" ::: "memory");
        issue(wr);
        compute(rd);
        wr = rd;
        rd = rd + STAGE == RING_BYTES ? 0 : rd + STAGE;
    }
#define TAIL_STEP(n) do { asm volatile("s_waitcnt vmcnt(" #n ") lgkmcnt(0)\n\ts_barrier" ::: "memory"); compute(rd); rd = rd + STAGE == RING_BYTES ? 0 : rd + STAGE; } while (0)
    TAIL_STEP(12); TAIL_STEP(9); TAIL_STEP(6); TAIL_STEP(3); TAIL_STEP(0);
#undef TAIL_STEP
    STAMP(3);

    float mx = 0.f;
#pragma unroll
    for (int i = 0; i < 4; ++i)
#pragma unroll
        for (int j = 0; j < 4; ++j)
#pragma unroll
            for (int r = 0; r < 4; ++r) mx = fmaxf(mx, fabsf(acc[i][j][r]));
    LAS unsigned* red = (LAS unsigned*)(lds + RING_BYTES);
    {
        const unsigned wm_ = wave_umax(__float_as_uint(mx));
        if (lane == 0) red[wid] = wm_;
    }
    asm volatile("s_waitcnt lgkmcnt(0)\n\ts_barrier" ::: "memory");
    const unsigned par = call & 1u;
    if (wid == 0) {
        const u32x4 a = *(const LAS u32x4*)red, b = *(const LAS u32x4*)(red + 4);
        const unsigned m = umax(umax(umax(a[0], a[1]), umax(a[2], a[3])), umax(umax(b[0], b[1]), umax(b[2], b[3])));
        if (lane == 0)
            __hip_atomic_store((unsigned long long*)(myws + WS_LMAX + par * (NWG * 8)) + bid, ((unsigned long long)call << 32) | m,
                               __ATOMIC_RELAXED, __HIP_MEMORY_SCOPE_AGENT);
    }
    STAMP(4);
    if (srcr == P.rank && tix < NRED) {
        char* const dst = P.ws[lane >> 3] + WS_GMAX + par * 1024 + (lane & 7) * 128 + P.rank * 8;
        const char* gp = myws + WS_LMAX + par * (NWG * 8) + lane * 16;
        const unsigned long long t0 = __builtin_amdgcn_s_memrealtime();
        unsigned polls = 0;
        for (;;) {
            u32x4 a, b;
            load16x2_sys(gp, a, b);
            if (__all(a[1] == call && a[3] == call && b[1] == call && b[3] == call)) {
                const unsigned m = wave_umax(umax(umax(a[0], a[2]), umax(b[0], b[2])));
                __hip_atomic_store((unsigned long long*)dst, ((unsigned long long)call << 32) | m, __ATOMIC_RELAXED, __HIP_MEMORY_SCOPE_SYSTEM);
                break;
            }
            if (give_up(tmo, call, t0, polls)) break;
        }
    }
    if (wid == 0) {
        const char* gp = myws + WS_GMAX + par * 1024 + (bid & 7) * 128 + (lane & 7) * 8;
        const unsigned long long t0 = __builtin_amdgcn_s_memrealtime();
        unsigned polls = 0, m = 0;
        bool dead = false;
        for (;;) {
            u32x2 a; unsigned t;
            asm volatile("global_load_dwordx2 %0, %2, off sc0 sc1\n\tglobal_load_dword %1, %3, off sc1\n\ts_waitcnt vmcnt(0)"
                         : "=&v"(a), "=&v"(t) : "v"(gp), "v"(tmo) : "memory");
            if (__builtin_amdgcn_readfirstlane(t) == call) { dead = true; break; }
            if (__all(a[1] == call)) { m = wave_umax(a[0]); break; }
            if (give_up(tmo, call, t0, polls)) { dead = true; break; }
        }
        if (lane == 0) { red[16] = m; red[17] = dead ? 1u : 0u; }
    }
    asm volatile("s_waitcnt lgkmcnt(0)\n\ts_barrier" ::: "memory");
    const unsigned gm = red[16], anydead = red[17];
    STAMP(5);

    const float amax = __uint_as_float(gm);
    const float inv = amax > 0.f ? 127.0f / amax : 0.f;
    char* op = P.out + (size_t)(srcr * MS + half * 256 + wm * 64 + (lane & 15)) * NL + n0 + wc * 64 + g * 16;
#pragma unroll
    for (int i = 0; i < 4; ++i) {
        unsigned d[4];
#pragma unroll
        for (int j = 0; j < 4; ++j) {
            unsigned b[4];
#pragma unroll
            for (int r = 0; r < 4; ++r) {
                const float t = __builtin_amdgcn_fmed3f(acc[i][j][r] * inv, -127.0f, 127.0f);
                b[r] = __float_as_uint(t + 12582912.0f);
            }
            const unsigned lo = __builtin_amdgcn_perm(b[1], b[0], 0x0c0c0400u), hi = __builtin_amdgcn_perm(b[3], b[2], 0x04000c0cu);
            d[j] = lo | hi;
        }
        const bool odd = g & 1;
        u32x4 o;
        o[0] = odd ? d[1] : d[0]; o[1] = odd ? d[0] : d[1]; o[2] = odd ? d[3] : d[2]; o[3] = odd ? d[2] : d[3];
        if (anydead) o = (u32x4){0x80808080u, 0x80808080u, 0x80808080u, 0x80808080u};
        if (AG_OST == 0) *(u32x4*)(op + (size_t)i * 16 * NL) = o;
        if (AG_OST == 1) __builtin_nontemporal_store(o, (u32x4*)(op + (size_t)i * 16 * NL));
        if (AG_OST == 2) store16_sys(op + (size_t)i * 16 * NL, o);
        if (AG_OST == 3) asm volatile("global_store_dwordx4 %0, %1, off sc1\n\ts_nop 1" :: "v"(op + (size_t)i * 16 * NL), "v"(o) : "memory");
    }
    if (tid == 0) __hip_atomic_store(ctr, call, __ATOMIC_RELAXED, __HIP_MEMORY_SCOPE_AGENT);
    STAMP(6);
}

}

extern "C" void kernel_launch(int rank, int nranks, ncclComm_t comm,
                              hipStream_t stream,
                              void* const* d_in, const int* in_sizes, int n_in,
                              void* d_out, int out_size,
                              const int* in_dtypes, int out_dtype,
                              void* const* d_out_all,
                              void* const* d_ws_all, size_t ws_bytes) {
    (void)comm; (void)in_dtypes; (void)out_dtype; (void)d_out_all;
    if (nranks != NR || n_in < 2 || rank < 0 || rank >= NR || in_sizes[0] != MS * KD || in_sizes[1] != KD * NL ||
        out_size != NR * MS * NL || ws_bytes < WS_END) {
        fprintf(stderr, "kernel_launch: unexpected problem shape\n");
        return;
    }
    Params P;
    P.x = (const char*)d_in[0];
    P.w = (const char*)d_in[1];
    P.out = (char*)d_out;
    for (int p = 0; p < NR; ++p) P.ws[p] = (char*)d_ws_all[p];
    P.myws = P.ws[rank];
    P.rank = rank;
    HIP_CHECK(hipFuncSetAttribute((const void*)ag_gemm_q8, hipFuncAttributeMaxDynamicSharedMemorySize, LDS_BYTES));
    hipLaunchKernelGGL(ag_gemm_q8, dim3(NWG), dim3(512), LDS_BYTES, stream, P);
    HIP_CHECK(hipGetLastError());
}
```

```cpp
#include <hip/hip_runtime.h>
#include <rccl/rccl.h>
#include <cstdio>

#define HIP_CHECK(expr) do { hipError_t _e = (expr); if (_e != hipSuccess) { \
    fprintf(stderr, "HIP error %s at %s:%d\n", hipGetErrorString(_e), __FILE__, __LINE__); \
    return; } } while (0)

namespace {

#define AG_MAP(rank) ((((0x4Du >> (rank)) & 1u) != 0u) ? 0x75416320u : 0x63207541u)
#define AG_SRC(b, map) ((int)(((map) >> (4 * ((b) & 7))) & 7u))

#define LAS __attribute__((address_space(3)))
typedef short s16x4 __attribute__((ext_vector_type(4)));
typedef short s16x8 __attribute__((ext_vector_type(8)));
typedef float f32x4 __attribute__((ext_vector_type(4)));
typedef unsigned u32x4 __attribute__((ext_vector_type(4)));
typedef unsigned u32x2 __attribute__((ext_vector_type(2)));

constexpr int NR = 8;
constexpr int MS = 512;
constexpr int KD = 4096;
constexpr int NL = 2048;
constexpr int NWG = 256;
constexpr int BK = 32, NSTEP = KD / BK;
constexpr int NSTAGE = 6;
constexpr int A_STAGE = 256 * BK * 2;
constexpr int B_STAGE = BK * 128 * 2;
constexpr int STAGE = A_STAGE + B_STAGE;
constexpr int RING_BYTES = NSTAGE * STAGE;
constexpr int LDS_BYTES = RING_BYTES + 256;
constexpr int NRED = 4;

constexpr size_t WS_XS = 0;
constexpr size_t WS_FLAG = (size_t)MS * KD * 2;
constexpr size_t WS_LMAX = WS_FLAG + 65536;
constexpr size_t WS_GMAX = WS_LMAX + 65536;
constexpr size_t WS_CTR = WS_GMAX + 65536;
constexpr size_t WS_TMO = WS_CTR + 4096;
constexpr size_t WS_XCC = WS_TMO + 4096;
constexpr size_t WS_DBG = WS_XCC + 4096;
constexpr size_t WS_END = WS_DBG + 65536;
constexpr unsigned long long DEADLINE = 50000000ull;

struct Params {
    const char* x;
    const char* w;
    char* out;
    char* myws;
    char* ws[NR];
    int rank;
    unsigned map;
};

__device__ __forceinline__ u32x2 load8_sys(const char* p) {
    u32x2 v;
    asm volatile("global_load_dwordx2 %0, %1, off sc0 sc1\n\ts_waitcnt vmcnt(0)" : "=&v"(v) : "v"(p) : "memory");
    return v;
}
__device__ __forceinline__ void load16x2_sys(const char* p, u32x4& a, u32x4& b) {
    asm volatile("global_load_dwordx4 %0, %2, off sc0 sc1\n\tglobal_load_dwordx4 %1, %2, off offset:1024 sc0 sc1\n\ts_waitcnt vmcnt(0)"
                 : "=&v"(a), "=&v"(b) : "v"(p) : "memory");
}
__device__ __forceinline__ void store16_sys(char* p, u32x4 v) {
    asm volatile("global_store_dwordx4 %0, %1, off sc0 sc1\n\ts_nop 1" :: "v"(p), "v"(v) : "memory");
}
__device__ __forceinline__ unsigned umax(unsigned a, unsigned b) { return a > b ? a : b; }
__device__ __forceinline__ unsigned wave_umax(unsigned v) {
    v = umax(v, (unsigned)__builtin_amdgcn_update_dpp(0, (int)v, 0xB1, 0xf, 0xf, true));
    v = umax(v, (unsigned)__builtin_amdgcn_update_dpp(0, (int)v, 0x4E, 0xf, 0xf, true));
    v = umax(v, (unsigned)__builtin_amdgcn_update_dpp(0, (int)v, 0x141, 0xf, 0xf, true));
    v = umax(v, (unsigned)__builtin_amdgcn_update_dpp(0, (int)v, 0x140, 0xf, 0xf, true));
    const unsigned a = __builtin_amdgcn_readlane(v, 0), b = __builtin_amdgcn_readlane(v, 16), c = __builtin_amdgcn_readlane(v, 32), d = __builtin_amdgcn_readlane(v, 48);
    return umax(umax(a, b), umax(c, d));
}
__device__ __forceinline__ bool give_up(unsigned* tmo, unsigned call, unsigned long long t0, unsigned& polls) {
    if ((++polls & 15u) != 0u) return false;
    const unsigned t = __builtin_amdgcn_readfirstlane(__hip_atomic_load(tmo, __ATOMIC_RELAXED, __HIP_MEMORY_SCOPE_AGENT));
    if (t == call || __builtin_amdgcn_s_memrealtime() - t0 > DEADLINE) {
        __hip_atomic_store(tmo, call, __ATOMIC_RELAXED, __HIP_MEMORY_SCOPE_AGENT);
        return true;
    }
    return false;
}

#define STAMP(i) do { } while (0)

__global__ __launch_bounds__(512, 2) void ag_gemm_q8(Params P) {
    extern __shared__ __attribute__((aligned(16))) char smem[];
    LAS char* lds = (LAS char*)smem;
    const int tid = threadIdx.x, lane = tid & 63, wid = __builtin_amdgcn_readfirstlane(tid >> 6);
    const int bid = blockIdx.x;
    char* const myws = P.myws;
    asm volatile("" :: "s"(P.x), "s"(P.w), "s"(P.out), "s"(P.myws), "s"(P.rank), "s"(P.map), "s"(P.ws[0]), "s"(P.ws[1]), "s"(P.ws[2]), "s"(P.ws[3]),
                 "s"(P.ws[4]), "s"(P.ws[5]), "s"(P.ws[6]), "s"(P.ws[7]));
    unsigned* const tmo = (unsigned*)(myws + WS_TMO);
    unsigned* const ctr = (unsigned*)(myws + WS_CTR) + bid;
    STAMP(0);
#define AG_INV 5
#define AG_OST 2

    unsigned call;
    unsigned xold = 0;
    if (wid < 4) {
        const int kt = bid >> 2, q = bid & 3;
        const int r0 = tid >> 3, c = tid & 7;
        const char* src = P.x + (size_t)(q * 128 + r0) * (KD * 2) + kt * 128 + c * 16;
        u32x4 v[4];
#pragma unroll
        for (int i = 0; i < 4; ++i) v[i] = *(const u32x4*)(src + (size_t)i * 32 * (KD * 2));
        call = __hip_atomic_load(ctr, __ATOMIC_RELAXED, __HIP_MEMORY_SCOPE_AGENT) + 1u;
        const int step = 2 * kt + (c >> 2), rt = (q & 1) * 128 + r0;
        char* dst = myws + WS_XS + (size_t)(step * 2 + (q >> 1)) * A_STAGE + (rt >> 4) * 1024 + ((c & 3) * 16 + (rt & 15)) * 16;
        asm volatile("" : "+v"(v[0]), "+v"(v[1]), "+v"(v[2]), "+v"(v[3]), "+v"(call));
#pragma unroll
        for (int i = 0; i < 4; ++i) store16_sys(dst + i * 2048, v[i]);
        asm volatile("s_waitcnt vmcnt(0)\n\ts_barrier" ::: "memory");
        if (tid == 64) {
            const unsigned foff = (unsigned)WS_FLAG + (unsigned)(P.rank * NWG + bid) * 4u;
#pragma unroll
            for (int p = 0; p < NR; ++p) asm volatile("global_store_dword %0, %1, %2 sc0 sc1" :: "v"(foff), "v"(call), "s"(P.ws[p]) : "memory");
        }
    } else {
        call = __hip_atomic_load(ctr, __ATOMIC_RELAXED, __HIP_MEMORY_SCOPE_AGENT) + 1u;
        asm volatile("" : "+v"(call));
        if (AG_INV == 1 && wid == 4) asm volatile("buffer_inv sc0 sc1" ::: "memory");
        asm volatile("s_barrier" ::: "memory");
        if (AG_INV >= 5 && wid == 7) {
            const unsigned xcc = (unsigned)__builtin_amdgcn_s_getreg((3 << 11) | 20) & 15u;
            if (lane == 0) xold = __hip_atomic_exchange((unsigned*)(myws + WS_XCC) + xcc * 32, call, __ATOMIC_RELAXED, __HIP_MEMORY_SCOPE_AGENT);
        }
    }
    call = __builtin_amdgcn_readfirstlane(call);
    STAMP(1);

    const int srcr = AG_SRC(bid, P.map);
    const int tix = bid >> 3, half = tix & 1, n0 = (tix >> 1) * 128;

    int ready = 0;
    const char* const fp = myws + WS_FLAG + (size_t)srcr * (NWG * 4) + half * 8 + lane * 16;
    auto wait_ready = [&](int need) {
        const unsigned long long t0 = __builtin_amdgcn_s_memrealtime();
        unsigned polls = 0;
        for (;;) {
            const u32x2 f = load8_sys(fp);
            const unsigned long long miss = ~__ballot(f[0] >= call && f[1] >= call);
            ready = miss ? 2 * __builtin_ctzll(miss) : NSTEP;
            if (ready >= need) break;
            if (give_up(tmo, call, t0, polls)) { ready = NSTEP; break; }
        }
    };
    const unsigned lds0 = (unsigned)(size_t)lds;
    const char* gA = P.ws[srcr] + WS_XS + (size_t)half * A_STAGE + wid * 2048;
    const char* gB = P.w + n0 * 2;
    const unsigned voffA = lane * 16;
    unsigned voffB;
    {
        const int row = 8 * (wid >> 1) + ((lane >> 2) & 7), chh = 2 * (wid & 1) + (lane >> 5);
        const int ch = chh * 4 + ((lane & 3) ^ ((row >> 2) & 3) ^ (chh & 1));
        voffB = (unsigned)(row * (NL * 2) + ch * 16);
    }
    const unsigned ldsWA = lds0 + wid * 2048, ldsWB = lds0 + A_STAGE + wid * 1024;
    auto issue = [&](int slot_off) {
        unsigned keep;
        asm volatile("s_mov_b32 %0, m0\n\ts_mov_b32 m0, %1\n\ts_nop 3\n\t"
                     "global_load_lds_dwordx4 %3, %4\n\t"
                     "global_load_lds_dwordx4 %3, %4 offset:1024\n\t"
                     "s_mov_b32 m0, %2\n\ts_nop 0\n\t"
                     "global_load_lds_dwordx4 %5, %6\n\t"
                     "s_mov_b32 m0, %0"
                     : "=&s"(keep) : "s"(ldsWA + slot_off), "s"(ldsWB + slot_off), "v"(voffA), "s"(gA), "v"(voffB), "s"(gB) : "memory");
        gA += 2 * A_STAGE; gB += BK * NL * 2;
    };

    const int wm = wid >> 1, wc = wid & 1;
    const int g = lane >> 4, q4 = (lane & 15) >> 2, p4 = lane & 3;
    const unsigned rdA = wm * 4096 + lane * 16;
    const unsigned rdB0 = A_STAGE + 2048 * g + 512 * (2 * wc + (p4 >> 1)) + 64 * q4 + 32 * ((p4 ^ g) & 1);
    unsigned rdB[2][2];
#pragma unroll
    for (int u = 0; u < 2; ++u)
#pragma unroll
        for (int v = 0; v < 2; ++v) rdB[u][v] = rdB0 + 16 * (u ^ (p4 >> 1)) + 8 * (v ^ (p4 & 1));

    f32x4 acc[4][4];
#pragma unroll
    for (int i = 0; i < 4; ++i)
#pragma unroll
        for (int j = 0; j < 4; ++j) acc[i][j] = (f32x4){0.f, 0.f, 0.f, 0.f};

    auto compute = [&](int slot_off) {
        s16x8 xf[4], wf[4];
#pragma unroll
        for (int j = 0; j < 4; ++j) {
            const s16x4 lo = __builtin_amdgcn_ds_read_tr16_b64_v4i16((LAS s16x4*)(lds + slot_off + rdB[j >> 1][j & 1]));
            const s16x4 hi = __builtin_amdgcn_ds_read_tr16_b64_v4i16((LAS s16x4*)(lds + slot_off + rdB[(j >> 1) ^ 1][j & 1] + 256));
            wf[j] = __builtin_shufflevector(lo, hi, 0, 1, 2, 3, 4, 5, 6, 7);
        }
#pragma unroll
        for (int i = 0; i < 4; ++i) xf[i] = *(const LAS s16x8*)(lds + slot_off + rdA + i * 1024);
#pragma unroll
        for (int i = 0; i < 4; ++i)
#pragma unroll
            for (int j = 0; j < 4; ++j) acc[i][j] = __builtin_amdgcn_mfma_f32_16x16x32_bf16(wf[j], xf[i], acc[i][j], 0, 0, 0);
    };

    asm volatile("" :: "s"(gA), "s"(gB), "v"(voffB));
    if (AG_INV == 2 && wid == 1) __builtin_amdgcn_fence(__ATOMIC_ACQUIRE, "");
    if (wid == 0) wait_ready(NSTAGE);
    if (AG_INV == 1 || AG_INV == 2) asm volatile("s_waitcnt vmcnt(0)" ::: "memory");
    asm volatile("s_barrier" ::: "memory");
    STAMP(2);

#pragma unroll
    for (int s = 0; s < NSTAGE - 1; ++s) issue(s * STAGE);
    int rd = 0, wr = (NSTAGE - 1) * STAGE;
    for (int s = 0; s < NSTEP - (NSTAGE - 1); ++s) {
        if (wid == 0 && ready < s + NSTAGE) wait_ready(s + NSTAGE);
        asm volatile("s_waitcnt vmcnt(12) lgkmcnt(0)\n\ts_barrier" ::: "memory");
        issue(wr);
        compute(rd);
        wr = rd;
        rd = rd + STAGE == RING_BYTES ? 0 : rd + STAGE;
    }
#define TAIL_STEP(n) do { asm volatile("s_waitcnt vmcnt(" #n ") lgkmcnt(0)\n\ts_barrier" ::: "memory"); compute(rd); rd = rd + STAGE == RING_BYTES ? 0 : rd + STAGE; } while (0)
    TAIL_STEP(12); TAIL_STEP(9); TAIL_STEP(6); TAIL_STEP(3); TAIL_STEP(0);
#undef TAIL_STEP
    STAMP(3);

    float mx = 0.f;
#pragma unroll
    for (int i = 0; i < 4; ++i)
#pragma unroll
        for (int j = 0; j < 4; ++j)
#pragma unroll
            for (int r = 0; r < 4; ++r) mx = fmaxf(mx, fabsf(acc[i][j][r]));
    LAS unsigned* red = (LAS unsigned*)(lds + RING_BYTES);
    {
        const unsigned wm_ = wave_umax(__float_as_uint(mx));
        if (lane == 0) red[wid] = wm_;
    }
    asm volatile("s_waitcnt lgkmcnt(0)\n\ts_barrier" ::: "memory");
    const unsigned par = call & 1u;
    if (wid == 0) {
        const u32x4 a = *(const LAS u32x4*)red, b = *(const LAS u32x4*)(red + 4);
        const unsigned m = umax(umax(umax(a[0], a[1]), umax(a[2], a[3])), umax(umax(b[0], b[1]), umax(b[2], b[3])));
        if (lane == 0)
            __hip_atomic_store((unsigned long long*)(myws + WS_LMAX + par * (NWG * 8)) + bid, ((unsigned long long)call << 32) | m,
                               __ATOMIC_RELAXED, __HIP_MEMORY_SCOPE_AGENT);
    }
    STAMP(4);
    if (wid == 1 && srcr == P.rank && tix < NRED) {
        char* const dst = P.ws[lane >> 3] + WS_GMAX + par * 1024 + (lane & 7) * 128 + P.rank * 8;
        const char* gp = myws + WS_LMAX + par * (NWG * 8) + lane * 16;
        const unsigned long long t0 = __builtin_amdgcn_s_memrealtime();
        unsigned polls = 0;
        for (;;) {
            u32x4 a, b;
            load16x2_sys(gp, a, b);
            if (__all(a[1] == call && a[3] == call && b[1] == call && b[3] == call)) {
                const unsigned m = wave_umax(umax(umax(a[0], a[2]), umax(b[0], b[2])));
                __hip_atomic_store((unsigned long long*)dst, ((unsigned long long)call << 32) | m, __ATOMIC_RELAXED, __HIP_MEMORY_SCOPE_SYSTEM);
                break;
            }
            if (give_up(tmo, call, t0, polls)) break;
        }
    }
    if (wid == 0) {
        const char* gp = myws + WS_GMAX + par * 1024 + (bid & 7) * 128 + (lane & 7) * 8;
        const unsigned long long t0 = __builtin_amdgcn_s_memrealtime();
        unsigned polls = 0, m = 0;
        bool dead = false;
        for (;;) {
            u32x2 a; unsigned t;
            asm volatile("global_load_dwordx2 %0, %2, off sc0 sc1\n\tglobal_load_dword %1, %3, off sc1\n\ts_waitcnt vmcnt(0)"
                         : "=&v"(a), "=&v"(t) : "v"(gp), "v"(tmo) : "memory");
            if (__builtin_amdgcn_readfirstlane(t) == call) { dead = true; break; }
            if (__all(a[1] == call)) { m = wave_umax(a[0]); break; }
            if (give_up(tmo, call, t0, polls)) { dead = true; break; }
        }
        if (lane == 0) { red[16] = m; red[17] = dead ? 1u : 0u; }
    }
    asm volatile("s_waitcnt lgkmcnt(0)\n\ts_barrier" ::: "memory");
    const unsigned gm = red[16], anydead = red[17];
    if (AG_INV == 6 && wid == 7 && __builtin_amdgcn_readfirstlane(xold) != call) asm volatile("buffer_inv sc0 sc1" ::: "memory");
    STAMP(5);

    const float amax = __uint_as_float(gm);
    const float inv = amax > 0.f ? 127.0f / amax : 0.f;
    char* op = P.out + (size_t)(srcr * MS + half * 256 + wm * 64 + (lane & 15)) * NL + n0 + wc * 64 + g * 16;
#pragma unroll
    for (int i = 0; i < 4; ++i) {
        unsigned d[4];
#pragma unroll
        for (int j = 0; j < 4; ++j) {
            unsigned b[4];
#pragma unroll
            for (int r = 0; r < 4; ++r) {
                b[r] = __float_as_uint(__builtin_fmaf(acc[i][j][r], inv, 12582912.0f));
            }
            const unsigned lo = __builtin_amdgcn_perm(b[1], b[0], 0x0c0c0400u), hi = __builtin_amdgcn_perm(b[3], b[2], 0x04000c0cu);
            d[j] = lo | hi;
        }
        const bool odd = g & 1;
        u32x4 o;
        o[0] = odd ? d[1] : d[0]; o[1] = odd ? d[0] : d[1]; o[2] = odd ? d[3] : d[2]; o[3] = odd ? d[2] : d[3];
        if (anydead) o = (u32x4){0x80808080u, 0x80808080u, 0x80808080u, 0x80808080u};
        if (AG_OST == 0) *(u32x4*)(op + (size_t)i * 16 * NL) = o;
        if (AG_OST == 1) __builtin_nontemporal_store(o, (u32x4*)(op + (size_t)i * 16 * NL));
        if (AG_OST == 2) store16_sys(op + (size_t)i * 16 * NL, o);
        if (AG_OST == 3) asm volatile("global_store_dwordx4 %0, %1, off sc1\n\ts_nop 1" :: "v"(op + (size_t)i * 16 * NL), "v"(o) : "memory");
    }
    if (tid == 0) __hip_atomic_store(ctr, call, __ATOMIC_RELAXED, __HIP_MEMORY_SCOPE_AGENT);
    if (AG_INV == 3 && wid == 7) asm volatile("buffer_inv sc0 sc1" ::: "memory");
    if (AG_INV == 5 && wid == 7 && __builtin_amdgcn_readfirstlane(xold) != call) asm volatile("buffer_inv sc0 sc1" ::: "memory");
    STAMP(6);
}

}

extern "C" void kernel_launch(int rank, int nranks, ncclComm_t comm,
                              hipStream_t stream,
                              void* const* d_in, const int* in_sizes, int n_in,
                              void* d_out, int out_size,
                              const int* in_dtypes, int out_dtype,
                              void* const* d_out_all,
                              void* const* d_ws_all, size_t ws_bytes) {
    (void)comm; (void)in_dtypes; (void)out_dtype; (void)d_out_all;
    if (nranks != NR || n_in < 2 || rank < 0 || rank >= NR || in_sizes[0] != MS * KD || in_sizes[1] != KD * NL ||
        out_size != NR * MS * NL || ws_bytes < WS_END) {
        fprintf(stderr, "kernel_launch: unexpected problem shape\n");
        return;
    }
    Params P;
    P.x = (const char*)d_in[0];
    P.w = (const char*)d_in[1];
    P.out = (char*)d_out;
    for (int p = 0; p < NR; ++p) P.ws[p] = (char*)d_ws_all[p];
    P.myws = P.ws[rank];
    P.rank = rank;
    P.map = AG_MAP(rank);
    HIP_CHECK(hipFuncSetAttribute((const void*)ag_gemm_q8, hipFuncAttributeMaxDynamicSharedMemorySize, LDS_BYTES));
    hipLaunchKernelGGL(ag_gemm_q8, dim3(NWG), dim3(512), LDS_BYTES, stream, P);
    HIP_CHECK(hipGetLastError());
}
```

```cpp
#include <hip/hip_runtime.h>
#include <rccl/rccl.h>
#include <cstdio>

#define HIP_CHECK(expr) do { hipError_t _e = (expr); if (_e != hipSuccess) { \
    fprintf(stderr, "HIP error %s at %s:%d\n", hipGetErrorString(_e), __FILE__, __LINE__); \
    return; } } while (0)

namespace {

#define AG_MAP(rank) ((((0x4Du >> (rank)) & 1u) != 0u) ? 0x75416320u : 0x63207541u)
#define AG_SRC(b, map) ((int)(((map) >> (4 * ((b) & 7))) & 7u))

#define LAS __attribute__((address_space(3)))
typedef short s16x4 __attribute__((ext_vector_type(4)));
typedef short s16x8 __attribute__((ext_vector_type(8)));
typedef float f32x4 __attribute__((ext_vector_type(4)));
typedef unsigned u32x4 __attribute__((ext_vector_type(4)));
typedef unsigned u32x2 __attribute__((ext_vector_type(2)));

constexpr int NR = 8;
constexpr int MS = 512;
constexpr int KD = 4096;
constexpr int NL = 2048;
constexpr int NWG = 256;
constexpr int BK = 32, NSTEP = KD / BK;
constexpr int NSTAGE = 6;
constexpr int A_STAGE = 256 * BK * 2;
constexpr int B_STAGE = BK * 128 * 2;
constexpr int STAGE = A_STAGE + B_STAGE;
constexpr int RING_BYTES = NSTAGE * STAGE;
constexpr int LDS_BYTES = RING_BYTES + 256;
constexpr int NRED = 4;

constexpr size_t WS_XS = 0;
constexpr size_t WS_FLAG = (size_t)MS * KD * 2;
constexpr size_t WS_LMAX = WS_FLAG + 65536;
constexpr size_t WS_GMAX = WS_LMAX + 65536;
constexpr size_t WS_CTR = WS_GMAX + 65536;
constexpr size_t WS_TMO = WS_CTR + 4096;
constexpr size_t WS_XCC = WS_TMO + 4096;
constexpr size_t WS_CLAIM = WS_XCC + 4096;
constexpr size_t WS_DBG = WS_CLAIM + 16384;
constexpr size_t WS_END = WS_DBG + 65536;
constexpr unsigned long long DEADLINE = 50000000ull;

struct Params {
    const char* x;
    const char* w;
    char* out;
    char* myws;
    char* ws[NR];
    int rank;
    unsigned map;
};

__device__ __forceinline__ u32x2 load8_sys(const char* p) {
    u32x2 v;
    asm volatile("global_load_dwordx2 %0, %1, off sc0 sc1\n\ts_waitcnt vmcnt(0)" : "=&v"(v) : "v"(p) : "memory");
    return v;
}
__device__ __forceinline__ void load16x2_sys(const char* p, u32x4& a, u32x4& b) {
    asm volatile("global_load_dwordx4 %0, %2, off sc0 sc1\n\tglobal_load_dwordx4 %1, %2, off offset:1024 sc0 sc1\n\ts_waitcnt vmcnt(0)"
                 : "=&v"(a), "=&v"(b) : "v"(p) : "memory");
}
__device__ __forceinline__ void store16_sys(char* p, u32x4 v) {
    asm volatile("global_store_dwordx4 %0, %1, off sc0 sc1\n\ts_nop 1" :: "v"(p), "v"(v) : "memory");
}
__device__ __forceinline__ unsigned umax(unsigned a, unsigned b) { return a > b ? a : b; }
__device__ __forceinline__ unsigned wave_umax(unsigned v) {
    v = umax(v, (unsigned)__builtin_amdgcn_update_dpp(0, (int)v, 0xB1, 0xf, 0xf, true));
    v = umax(v, (unsigned)__builtin_amdgcn_update_dpp(0, (int)v, 0x4E, 0xf, 0xf, true));
    v = umax(v, (unsigned)__builtin_amdgcn_update_dpp(0, (int)v, 0x141, 0xf, 0xf, true));
    v = umax(v, (unsigned)__builtin_amdgcn_update_dpp(0, (int)v, 0x140, 0xf, 0xf, true));
    const unsigned a = __builtin_amdgcn_readlane(v, 0), b = __builtin_amdgcn_readlane(v, 16), c = __builtin_amdgcn_readlane(v, 32), d = __builtin_amdgcn_readlane(v, 48);
    return umax(umax(a, b), umax(c, d));
}
__device__ __forceinline__ bool give_up(unsigned* tmo, unsigned call, unsigned long long t0, unsigned& polls) {
    if ((++polls & 15u) != 0u) return false;
    const unsigned t = __builtin_amdgcn_readfirstlane(__hip_atomic_load(tmo, __ATOMIC_RELAXED, __HIP_MEMORY_SCOPE_AGENT));
    if (t == call || __builtin_amdgcn_s_memrealtime() - t0 > DEADLINE) {
        __hip_atomic_store(tmo, call, __ATOMIC_RELAXED, __HIP_MEMORY_SCOPE_AGENT);
        return true;
    }
    return false;
}

#define STAMP(i) do { } while (0)

__global__ __launch_bounds__(512, 2) void ag_gemm_q8(Params P) {
    extern __shared__ __attribute__((aligned(16))) char smem[];
    LAS char* lds = (LAS char*)smem;
    const int tid = threadIdx.x, lane = tid & 63, wid = __builtin_amdgcn_readfirstlane(tid >> 6);
    const int bid = blockIdx.x;
    char* const myws = P.myws;
    asm volatile("" :: "s"(P.x), "s"(P.w), "s"(P.out), "s"(P.myws), "s"(P.rank), "s"(P.map), "s"(P.ws[0]), "s"(P.ws[1]), "s"(P.ws[2]), "s"(P.ws[3]),
                 "s"(P.ws[4]), "s"(P.ws[5]), "s"(P.ws[6]), "s"(P.ws[7]));
    unsigned* const tmo = (unsigned*)(myws + WS_TMO);
    unsigned* const ctr = (unsigned*)(myws + WS_CTR) + bid;
    STAMP(0);
#define AG_INV 5
#define AG_OST 2

    unsigned call;
    unsigned xold = 0;
    if (wid < 4) {
        const int kt = bid >> 2, q = bid & 3;
        const int r0 = tid >> 3, c = tid & 7;
        const char* src = P.x + (size_t)(q * 128 + r0) * (KD * 2) + kt * 128 + c * 16;
        u32x4 v[4];
#pragma unroll
        for (int i = 0; i < 4; ++i) v[i] = *(const u32x4*)(src + (size_t)i * 32 * (KD * 2));
        call = __hip_atomic_load(ctr, __ATOMIC_RELAXED, __HIP_MEMORY_SCOPE_AGENT) + 1u;
        const int step = 2 * kt + (c >> 2), rt = (q & 1) * 128 + r0;
        char* dst = myws + WS_XS + (size_t)(step * 2 + (q >> 1)) * A_STAGE + (rt >> 4) * 1024 + ((c & 3) * 16 + (rt & 15)) * 16;
        asm volatile("" : "+v"(v[0]), "+v"(v[1]), "+v"(v[2]), "+v"(v[3]), "+v"(call));
#pragma unroll
        for (int i = 0; i < 4; ++i) store16_sys(dst + i * 2048, v[i]);
        asm volatile("s_waitcnt vmcnt(0)\n\ts_barrier" ::: "memory");
        if (tid == 64) {
            const unsigned foff = (unsigned)WS_FLAG + (unsigned)(P.rank * NWG + bid) * 4u;
#pragma unroll
            for (int p = 0; p < NR; ++p) asm volatile("global_store_dword %0, %1, %2 sc0 sc1" :: "v"(foff), "v"(call), "s"(P.ws[p]) : "memory");
        }
    } else {
        call = __hip_atomic_load(ctr, __ATOMIC_RELAXED, __HIP_MEMORY_SCOPE_AGENT) + 1u;
        asm volatile("" : "+v"(call));
        if (wid == 4) {
            const unsigned xcc = (unsigned)__builtin_amdgcn_s_getreg((3 << 11) | 20) & 7u;
            const unsigned ucall = __builtin_amdgcn_readfirstlane(call);
            unsigned t = ((unsigned)bid & ~7u) | xcc;
#pragma unroll 1
            for (int i = 0; i < NWG; ++i) {
                unsigned old = 0;
                if (lane == 0) old = __hip_atomic_exchange((unsigned*)(myws + WS_CLAIM) + t * 16, ucall, __ATOMIC_RELAXED, __HIP_MEMORY_SCOPE_AGENT);
                if (__builtin_amdgcn_readfirstlane(old) != ucall) break;
                t = (t + 1) & (NWG - 1);
            }
            if (lane == 0) *(LAS unsigned*)(lds + RING_BYTES + 128) = t;
        }
        asm volatile("s_waitcnt lgkmcnt(0)\n\ts_barrier" ::: "memory");
        if (AG_INV >= 5 && wid == 7) {
            const unsigned xcc = (unsigned)__builtin_amdgcn_s_getreg((3 << 11) | 20) & 15u;
            if (lane == 0) xold = __hip_atomic_exchange((unsigned*)(myws + WS_XCC) + xcc * 32, call, __ATOMIC_RELAXED, __HIP_MEMORY_SCOPE_AGENT);
        }
    }
    call = __builtin_amdgcn_readfirstlane(call);
    STAMP(1);

    const int tile = __builtin_amdgcn_readfirstlane(*(const volatile LAS unsigned*)(lds + RING_BYTES + 128));
    const int srcr = AG_SRC(tile, P.map);
    const int tix = tile >> 3, half = tix & 1, n0 = (tix >> 1) * 128;

    int ready = 0;
    const char* const fp = myws + WS_FLAG + (size_t)srcr * (NWG * 4) + half * 8 + lane * 16;
    auto wait_ready = [&](int need) {
        const unsigned long long t0 = __builtin_amdgcn_s_memrealtime();
        unsigned polls = 0;
        for (;;) {
            const u32x2 f = load8_sys(fp);
            const unsigned long long miss = ~__ballot(f[0] >= call && f[1] >= call);
            ready = miss ? 2 * __builtin_ctzll(miss) : NSTEP;
            if (ready >= need) break;
            if (give_up(tmo, call, t0, polls)) { ready = NSTEP; break; }
        }
    };
    const unsigned lds0 = (unsigned)(size_t)lds;
    const char* gA = P.ws[srcr] + WS_XS + (size_t)half * A_STAGE + wid * 2048;
    const char* gB = P.w + n0 * 2;
    const unsigned voffA = lane * 16;
    unsigned voffB;
    {
        const int row = 8 * (wid >> 1) + ((lane >> 2) & 7), chh = 2 * (wid & 1) + (lane >> 5);
        const int ch = chh * 4 + ((lane & 3) ^ ((row >> 2) & 3) ^ (chh & 1));
        voffB = (unsigned)(row * (NL * 2) + ch * 16);
    }
    const unsigned ldsWA = lds0 + wid * 2048, ldsWB = lds0 + A_STAGE + wid * 1024;
    auto issue = [&](int slot_off) {
        unsigned keep;
        asm volatile("s_mov_b32 %0, m0\n\ts_mov_b32 m0, %1\n\ts_nop 3\n\t"
                     "global_load_lds_dwordx4 %3, %4\n\t"
                     "global_load_lds_dwordx4 %3, %4 offset:1024\n\t"
                     "s_mov_b32 m0, %2\n\ts_nop 0\n\t"
                     "global_load_lds_dwordx4 %5, %6\n\t"
                     "s_mov_b32 m0, %0"
                     : "=&s"(keep) : "s"(ldsWA + slot_off), "s"(ldsWB + slot_off), "v"(voffA), "s"(gA), "v"(voffB), "s"(gB) : "memory");
        gA += 2 * A_STAGE; gB += BK * NL * 2;
    };

    const int wm = wid >> 1, wc = wid & 1;
    const int g = lane >> 4, q4 = (lane & 15) >> 2, p4 = lane & 3;
    const unsigned rdA = wm * 4096 + lane * 16;
    const unsigned rdB0 = A_STAGE + 2048 * g + 512 * (2 * wc + (p4 >> 1)) + 64 * q4 + 32 * ((p4 ^ g) & 1);
    unsigned rdB[2][2];
#pragma unroll
    for (int u = 0; u < 2; ++u)
#pragma unroll
        for (int v = 0; v < 2; ++v) rdB[u][v] = rdB0 + 16 * (u ^ (p4 >> 1)) + 8 * (v ^ (p4 & 1));

    f32x4 acc[4][4];
#pragma unroll
    for (int i = 0; i < 4; ++i)
#pragma unroll
        for (int j = 0; j < 4; ++j) acc[i][j] = (f32x4){0.f, 0.f, 0.f, 0.f};

    auto compute = [&](int slot_off) {
        s16x8 xf[4], wf[4];
#pragma unroll
        for (int j = 0; j < 4; ++j) {
            const s16x4 lo = __builtin_amdgcn_ds_read_tr16_b64_v4i16((LAS s16x4*)(lds + slot_off + rdB[j >> 1][j & 1]));
            const s16x4 hi = __builtin_amdgcn_ds_read_tr16_b64_v4i16((LAS s16x4*)(lds + slot_off + rdB[(j >> 1) ^ 1][j & 1] + 256));
            wf[j] = __builtin_shufflevector(lo, hi, 0, 1, 2, 3, 4, 5, 6, 7);
        }
#pragma unroll
        for (int i = 0; i < 4; ++i) xf[i] = *(const LAS s16x8*)(lds + slot_off + rdA + i * 1024);
#pragma unroll
        for (int i = 0; i < 4; ++i)
#pragma unroll
            for (int j = 0; j < 4; ++j) acc[i][j] = __builtin_amdgcn_mfma_f32_16x16x32_bf16(wf[j], xf[i], acc[i][j], 0, 0, 0);
    };

    asm volatile("" :: "s"(gA), "s"(gB), "v"(voffB));
    if (wid == 0) wait_ready(NSTAGE);
    asm volatile("s_barrier" ::: "memory");
    STAMP(2);

#pragma unroll
    for (int s = 0; s < NSTAGE - 1; ++s) issue(s * STAGE);
    int rd = 0, wr = (NSTAGE - 1) * STAGE;
    for (int s = 0; s < NSTEP - (NSTAGE - 1); ++s) {
        if (wid == 0 && ready < s + NSTAGE) wait_ready(s + NSTAGE);
        asm volatile("s_waitcnt vmcnt(12) lgkmcnt(0)\n\ts_barrier" ::: "memory");
        issue(wr);
        compute(rd);
        wr = rd;
        rd = rd + STAGE == RING_BYTES ? 0 : rd + STAGE;
    }
#define TAIL_STEP(n) do { asm volatile("s_waitcnt vmcnt(" #n ") lgkmcnt(0)\n\ts_barrier" ::: "memory"); compute(rd); rd = rd + STAGE == RING_BYTES ? 0 : rd + STAGE; } while (0)
    TAIL_STEP(12); TAIL_STEP(9); TAIL_STEP(6); TAIL_STEP(3); TAIL_STEP(0);
#undef TAIL_STEP
    STAMP(3);

    float mx = 0.f;
#pragma unroll
    for (int i = 0; i < 4; ++i)
#pragma unroll
        for (int j = 0; j < 4; ++j)
#pragma unroll
            for (int r = 0; r < 4; ++r) mx = fmaxf(mx, fabsf(acc[i][j][r]));
    LAS unsigned* red = (LAS unsigned*)(lds + RING_BYTES);
    {
        const unsigned wm_ = wave_umax(__float_as_uint(mx));
        if (lane == 0) red[wid] = wm_;
    }
    asm volatile("s_waitcnt lgkmcnt(0)\n\ts_barrier" ::: "memory");
    const unsigned par = call & 1u;
    if (wid == 0) {
        const u32x4 a = *(const LAS u32x4*)red, b = *(const LAS u32x4*)(red + 4);
        const unsigned m = umax(umax(umax(a[0], a[1]), umax(a[2], a[3])), umax(umax(b[0], b[1]), umax(b[2], b[3])));
        if (lane == 0)
            __hip_atomic_store((unsigned long long*)(myws + WS_LMAX + par * (NWG * 8)) + bid, ((unsigned long long)call << 32) | m,
                               __ATOMIC_RELAXED, __HIP_MEMORY_SCOPE_AGENT);
    }
    STAMP(4);
    if (wid == 1 && srcr == P.rank && tix < NRED) {
        char* const dst = P.ws[lane >> 3] + WS_GMAX + par * 1024 + (lane & 7) * 128 + P.rank * 8;
        const char* gp = myws + WS_LMAX + par * (NWG * 8) + lane * 16;
        const unsigned long long t0 = __builtin_amdgcn_s_memrealtime();
        unsigned polls = 0;
        for (;;) {
            u32x4 a, b;
            load16x2_sys(gp, a, b);
            if (__all(a[1] == call && a[3] == call && b[1] == call && b[3] == call)) {
                const unsigned m = wave_umax(umax(umax(a[0], a[2]), umax(b[0], b[2])));
                __hip_atomic_store((unsigned long long*)dst, ((unsigned long long)call << 32) | m, __ATOMIC_RELAXED, __HIP_MEMORY_SCOPE_SYSTEM);
                break;
            }
            if (give_up(tmo, call, t0, polls)) break;
        }
    }
    if (wid == 0) {
        const char* gp = myws + WS_GMAX + par * 1024 + (tile & 7) * 128 + (lane & 7) * 8;
        const unsigned long long t0 = __builtin_amdgcn_s_memrealtime();
        unsigned polls = 0, m = 0;
        bool dead = false;
        for (;;) {
            u32x2 a; unsigned t;
            asm volatile("global_load_dwordx2 %0, %2, off sc0 sc1\n\tglobal_load_dword %1, %3, off sc1\n\ts_waitcnt vmcnt(0)"
                         : "=&v"(a), "=&v"(t) : "v"(gp), "v"(tmo) : "memory");
            if (__builtin_amdgcn_readfirstlane(t) == call) { dead = true; break; }
            if (__all(a[1] == call)) { m = wave_umax(a[0]); break; }
            if (give_up(tmo, call, t0, polls)) { dead = true; break; }
        }
        if (lane == 0) { red[16] = m; red[17] = dead ? 1u : 0u; }
    }
    asm volatile("s_waitcnt lgkmcnt(0)\n\ts_barrier" ::: "memory");
    const unsigned gm = red[16], anydead = red[17];
    STAMP(5);

    const float amax = __uint_as_float(gm);
    const float inv = amax > 0.f ? 127.0f / amax : 0.f;
    char* op = P.out + (size_t)(srcr * MS + half * 256 + wm * 64 + (lane & 15)) * NL + n0 + wc * 64 + g * 16;
#pragma unroll
    for (int i = 0; i < 4; ++i) {
        unsigned d[4];
#pragma unroll
        for (int j = 0; j < 4; ++j) {
            unsigned b[4];
#pragma unroll
            for (int r = 0; r < 4; ++r) {
                b[r] = __float_as_uint(__builtin_fmaf(acc[i][j][r], inv, 12582912.0f));
            }
            const unsigned lo = __builtin_amdgcn_perm(b[1], b[0], 0x0c0c0400u), hi = __builtin_amdgcn_perm(b[3], b[2], 0x04000c0cu);
            d[j] = lo | hi;
        }
        const bool odd = g & 1;
        u32x4 o;
        o[0] = odd ? d[1] : d[0]; o[1] = odd ? d[0] : d[1]; o[2] = odd ? d[3] : d[2]; o[3] = odd ? d[2] : d[3];
        if (anydead) o = (u32x4){0x80808080u, 0x80808080u, 0x80808080u, 0x80808080u};
        if (AG_OST == 0) *(u32x4*)(op + (size_t)i * 16 * NL) = o;
        if (AG_OST == 1) __builtin_nontemporal_store(o, (u32x4*)(op + (size_t)i * 16 * NL));
        if (AG_OST == 2) store16_sys(op + (size_t)i * 16 * NL, o);
        if (AG_OST == 3) asm volatile("global_store_dwordx4 %0, %1, off sc1\n\ts_nop 1" :: "v"(op + (size_t)i * 16 * NL), "v"(o) : "memory");
    }
    if (tid == 0) __hip_atomic_store(ctr, call, __ATOMIC_RELAXED, __HIP_MEMORY_SCOPE_AGENT);
    if (AG_INV == 5 && wid == 7 && __builtin_amdgcn_readfirstlane(xold) != call) asm volatile("buffer_inv sc0 sc1" ::: "memory");
    STAMP(6);
}

}

extern "C" void kernel_launch(int rank, int nranks, ncclComm_t comm,
                              hipStream_t stream,
                              void* const* d_in, const int* in_sizes, int n_in,
                              void* d_out, int out_size,
                              const int* in_dtypes, int out_dtype,
                              void* const* d_out_all,
                              void* const* d_ws_all, size_t ws_bytes) {
    (void)comm; (void)in_dtypes; (void)out_dtype; (void)d_out_all;
    if (nranks != NR || n_in < 2 || rank < 0 || rank >= NR || in_sizes[0] != MS * KD || in_sizes[1] != KD * NL ||
        out_size != NR * MS * NL || ws_bytes < WS_END) {
        fprintf(stderr, "kernel_launch: unexpected problem shape\n");
        return;
    }
    Params P;
    P.x = (const char*)d_in[0];
    P.w = (const char*)d_in[1];
    P.out = (char*)d_out;
    for (int p = 0; p < NR; ++p) P.ws[p] = (char*)d_ws_all[p];
    P.myws = P.ws[rank];
    P.rank = rank;
    P.map = AG_MAP(rank);
    HIP_CHECK(hipFuncSetAttribute((const void*)ag_gemm_q8, hipFuncAttributeMaxDynamicSharedMemorySize, LDS_BYTES));
    hipLaunchKernelGGL(ag_gemm_q8, dim3(NWG), dim3(512), LDS_BYTES, stream, P);
    HIP_CHECK(hipGetLastError());
}
```

```cpp
#include <hip/hip_runtime.h>
#include <rccl/rccl.h>
#include <cstdio>

#define HIP_CHECK(expr) do { hipError_t _e = (expr); if (_e != hipSuccess) { \
    fprintf(stderr, "HIP error %s at %s:%d\n", hipGetErrorString(_e), __FILE__, __LINE__); \
    return; } } while (0)

namespace {

#define AG_MAP(rank) ((((0x4Du >> (rank)) & 1u) != 0u) ? 0x75416320u : 0x63207541u)
#define AG_SRC(g, map) ((int)(((map) >> (4 * ((g) & 7))) & 7u))

#define LAS __attribute__((address_space(3)))
typedef short s16x4 __attribute__((ext_vector_type(4)));
typedef short s16x8 __attribute__((ext_vector_type(8)));
typedef float f32x4 __attribute__((ext_vector_type(4)));
typedef unsigned u32x4 __attribute__((ext_vector_type(4)));
typedef unsigned u32x2 __attribute__((ext_vector_type(2)));

constexpr int NR = 8;
constexpr int MS = 512;
constexpr int KD = 4096;
constexpr int NL = 2048;
constexpr int NWG = 256;
constexpr int BK = 32, NSTEP = KD / BK;
constexpr int NSTAGE = 4;
constexpr int A_STAGE = 256 * BK * 2;
constexpr int B_STAGE = BK * 128 * 2;
constexpr int LDS_A = 0, LDS_B = NSTAGE * A_STAGE, LDS_X = LDS_B + NSTAGE * B_STAGE;
constexpr int LDS_BYTES = LDS_X + 256 + 1024;
constexpr int NRED = 2;
constexpr int NREP = 8;

constexpr size_t WS_XS = 0;
constexpr size_t WS_FLAG = (size_t)MS * KD * 2;
constexpr size_t WS_LMAX = WS_FLAG + 65536;
constexpr size_t WS_GMAX = WS_LMAX + 65536;
constexpr size_t WS_CTR = WS_GMAX + 65536;
constexpr size_t WS_TMO = WS_CTR + 4096;
constexpr size_t WS_XCC = WS_TMO + 4096;
constexpr size_t WS_CLAIM = WS_XCC + 4096;
constexpr size_t WS_DBG = WS_CLAIM + 16384;
constexpr size_t WS_END = WS_DBG + 65536;
constexpr unsigned long long DEADLINE = 50000000ull;

template <int N> struct IC { static constexpr int value = N; };

struct Params {
    const char* x;
    const char* w;
    char* out;
    char* myws;
    char* ws[NR];
    int rank;
    unsigned map;
};

__device__ __forceinline__ u32x2 load8_sys(const char* p) {
    u32x2 v;
    asm volatile("global_load_dwordx2 %0, %1, off sc0 sc1\n\ts_waitcnt vmcnt(0)" : "=&v"(v) : "v"(p) : "memory");
    return v;
}
__device__ __forceinline__ void load16x2_sys(const char* p, u32x4& a, u32x4& b) {
    asm volatile("global_load_dwordx4 %0, %2, off sc0 sc1\n\tglobal_load_dwordx4 %1, %2, off offset:1024 sc0 sc1\n\ts_waitcnt vmcnt(0)"
                 : "=&v"(a), "=&v"(b) : "v"(p) : "memory");
}
__device__ __forceinline__ void dma16_sys(unsigned lds_addr, const char* p) {
    unsigned keep;
    asm volatile("s_mov_b32 %0, m0\n\ts_mov_b32 m0, %1\n\ts_nop 0\n\tglobal_load_lds_dwordx4 %2, off sc0 sc1\n\ts_mov_b32 m0, %0"
                 : "=&s"(keep) : "s"(lds_addr), "v"(p) : "memory");
}
#define AG_APOL ""
#define AG_BPOL ""
template <int SLOT>
__device__ __forceinline__ void stage_dma(unsigned ldsA, unsigned ldsB, unsigned va, const char* ga, unsigned vb, const char* gb) {
    asm volatile("s_add_i32 m0, %0, %6\n\ts_nop 0\n\t"
                 "global_load_lds_dwordx4 %2, %3" AG_APOL "\n\t"
                 "global_load_lds_dwordx4 %2, %3 offset:1024" AG_APOL "\n\t"
                 "s_add_i32 m0, %1, %7\n\ts_nop 0\n\t"
                 "global_load_lds_dwordx4 %4, %5" AG_BPOL
                 :: "s"(ldsA), "s"(ldsB), "v"(va), "s"(ga), "v"(vb), "s"(gb), "n"(SLOT * A_STAGE), "n"(SLOT * B_STAGE) : "memory", "scc");
}
template <int OFF, int IMM>
__device__ __forceinline__ void dma1(unsigned ldsbase, unsigned v, const char* gp) {
    asm volatile("s_add_i32 m0, %0, %3\n\ts_nop 0\n\tglobal_load_lds_dwordx4 %1, %2 offset:%4" :: "s"(ldsbase), "v"(v), "s"(gp), "n"(OFF), "n"(IMM) : "memory", "scc");
}
__device__ __forceinline__ void store16_sys(char* p, u32x4 v) {
    asm volatile("global_store_dwordx4 %0, %1, off sc0 sc1\n\ts_nop 1" :: "v"(p), "v"(v) : "memory");
}
__device__ __forceinline__ unsigned umax(unsigned a, unsigned b) { return a > b ? a : b; }
__device__ __forceinline__ unsigned wave_umax(unsigned v) {
    v = umax(v, (unsigned)__builtin_amdgcn_update_dpp(0, (int)v, 0xB1, 0xf, 0xf, true));
    v = umax(v, (unsigned)__builtin_amdgcn_update_dpp(0, (int)v, 0x4E, 0xf, 0xf, true));
    v = umax(v, (unsigned)__builtin_amdgcn_update_dpp(0, (int)v, 0x141, 0xf, 0xf, true));
    v = umax(v, (unsigned)__builtin_amdgcn_update_dpp(0, (int)v, 0x140, 0xf, 0xf, true));
    const unsigned a = __builtin_amdgcn_readlane(v, 0), b = __builtin_amdgcn_readlane(v, 16), c = __builtin_amdgcn_readlane(v, 32), d = __builtin_amdgcn_readlane(v, 48);
    return umax(umax(a, b), umax(c, d));
}
__device__ __forceinline__ bool give_up(unsigned* tmo, unsigned call, unsigned long long t0, unsigned& polls) {
    if ((++polls & 15u) != 0u) return false;
    const unsigned t = __builtin_amdgcn_readfirstlane(__hip_atomic_load(tmo, __ATOMIC_RELAXED, __HIP_MEMORY_SCOPE_AGENT));
    if (t == call || __builtin_amdgcn_s_memrealtime() - t0 > DEADLINE) {
        __hip_atomic_store(tmo, call, __ATOMIC_RELAXED, __HIP_MEMORY_SCOPE_AGENT);
        return true;
    }
    return false;
}

#define STAMP(i) do { } while (0)
#define AG_IPOS 7
#define AG_SCHED 2
#define AG_XSLEEP 0
#define AG_ABL 0

__global__ __launch_bounds__(512, 2) void ag_gemm_q8(Params P) {
    extern __shared__ __attribute__((aligned(16))) char smem[];
    LAS char* lds = (LAS char*)smem;
    const int tid = threadIdx.x, lane = tid & 63, wid = __builtin_amdgcn_readfirstlane(tid >> 6);
    const int bid = blockIdx.x;
    char* const myws = P.myws;
    asm volatile("" :: "s"(P.x), "s"(P.w), "s"(P.out), "s"(P.myws), "s"(P.rank), "s"(P.map), "s"(P.ws[0]), "s"(P.ws[1]), "s"(P.ws[2]),
                 "s"(P.ws[3]), "s"(P.ws[4]), "s"(P.ws[5]), "s"(P.ws[6]), "s"(P.ws[7]));
    unsigned* const tmo = (unsigned*)(myws + WS_TMO);
    unsigned* const ctr = (unsigned*)(myws + WS_CTR) + bid;
    LAS unsigned* const red = (LAS unsigned*)(lds + LDS_X);
    STAMP(0);

    unsigned call;
    unsigned xold = 0;
    if (wid < 4) {
        const int kt = bid >> 2, q = bid & 3;
        const int r0 = tid >> 3, c = tid & 7;
        const char* src = P.x + (size_t)(q * 128 + r0) * (KD * 2) + kt * 128 + c * 16;
        u32x4 v[4];
#pragma unroll
        for (int i = 0; i < 4; ++i) v[i] = *(const u32x4*)(src + (size_t)i * 32 * (KD * 2));
        call = __hip_atomic_load(ctr, __ATOMIC_RELAXED, __HIP_MEMORY_SCOPE_AGENT) + 1u;
        const int step = 2 * kt + (c >> 2), rt = (q & 1) * 128 + r0;
        char* dst = myws + WS_XS + (size_t)(step * 2 + (q >> 1)) * A_STAGE + (rt >> 4) * 1024 + ((c & 3) * 16 + (rt & 15)) * 16;
        asm volatile("" : "+v"(v[0]), "+v"(v[1]), "+v"(v[2]), "+v"(v[3]), "+v"(call));
#pragma unroll
        for (int i = 0; i < 4; ++i) store16_sys(dst + i * 2048, v[i]);
        asm volatile("s_waitcnt vmcnt(0)\n\ts_barrier" ::: "memory");
        if (tid == 64) {
            const unsigned foff = (unsigned)WS_FLAG + (unsigned)(P.rank * NWG + bid) * 4u;
#pragma unroll
            for (int p = 0; p < NR; ++p) asm volatile("global_store_dword %0, %1, %2 sc0 sc1" :: "v"(foff), "v"(call), "s"(P.ws[p]) : "memory");
        }
    } else {
        call = __hip_atomic_load(ctr, __ATOMIC_RELAXED, __HIP_MEMORY_SCOPE_AGENT) + 1u;
        asm volatile("" : "+v"(call));
        const unsigned xcc = (unsigned)__builtin_amdgcn_s_getreg((3 << 11) | 20);
        if (wid == 4) {
            const unsigned ucall = __builtin_amdgcn_readfirstlane(call);
            unsigned t = ((unsigned)bid & ~7u) | (xcc & 7u);
#pragma unroll 1
            for (int i = 0; i < NWG; ++i) {
                unsigned old = 0;
                if (lane == 0) old = __hip_atomic_exchange((unsigned*)(myws + WS_CLAIM) + t * 16, ucall, __ATOMIC_RELAXED, __HIP_MEMORY_SCOPE_AGENT);
                if (__builtin_amdgcn_readfirstlane(old) != ucall) break;
                t = (t + 1) & (NWG - 1);
            }
            if (lane == 0) red[32] = t;
        }
        asm volatile("s_waitcnt lgkmcnt(0)\n\ts_barrier" ::: "memory");
        if (wid == 7 && lane == 0)
            xold = __hip_atomic_exchange((unsigned*)(myws + WS_XCC) + xcc * 32, call, __ATOMIC_RELAXED, __HIP_MEMORY_SCOPE_AGENT);
    }
    call = __builtin_amdgcn_readfirstlane(call);
    STAMP(1);

    const int tile = __builtin_amdgcn_readfirstlane(*(const volatile LAS unsigned*)(red + 32));
    const int srcr = AG_SRC(tile, P.map);
    const int tix = tile >> 3, half = tix & 1, n0 = (tix >> 1) * 128;

    const unsigned lds0 = (unsigned)(size_t)lds;
    int ready = 0;
    const char* const fp = myws + WS_FLAG + (size_t)srcr * (NWG * 4);
    auto ready_of = [&](u32x2 f) {
        const unsigned long long miss = ~__ballot(f[0] >= call && f[1] >= call);
        return miss ? 2 * (int)__builtin_ctzll(miss) : NSTEP;
    };
    auto wait_ready = [&](int need, int tiles) {
        const unsigned long long t0 = __builtin_amdgcn_s_memrealtime();
        unsigned polls = 0;
        for (;;) {
            ready = ready_of(load8_sys(fp + (lane & (tiles - 1)) * 16 + half * 8));
            if (ready > 2 * tiles) ready = 2 * tiles;
            if (ready >= need) break;
            if (give_up(tmo, call, t0, polls)) { ready = NSTEP; break; }
        }
    };

    const char* gA = (AG_ABL ? myws : P.ws[srcr]) + WS_XS + (size_t)half * A_STAGE + wid * 2048;
    const char* gB = P.w + n0 * 2;
    unsigned voffA[NSTAGE], voffB[NSTAGE];
    {
        const int row = 8 * (wid >> 1) + ((lane >> 2) & 7), chh = 2 * (wid & 1) + (lane >> 5);
        const int ch = chh * 4 + ((lane & 3) ^ ((row >> 2) & 3) ^ (chh & 1));
#pragma unroll
        for (int k = 0; k < NSTAGE; ++k) {
            voffA[k] = (unsigned)(lane * 16 + k * 2 * A_STAGE);
            voffB[k] = (unsigned)(row * (NL * 2) + ch * 16 + k * BK * NL * 2);
        }
    }
    const unsigned ldsWA = lds0 + LDS_A + wid * 2048, ldsWB = lds0 + LDS_B + wid * 1024;
    auto issue = [&](auto K, auto SLOT) {
        stage_dma<decltype(SLOT)::value>(ldsWA, ldsWB, voffA[decltype(K)::value], gA, voffB[decltype(K)::value], gB);
    };

    const int wm = wid >> 1, wc = wid & 1;
    const int g = lane >> 4, q4 = (lane & 15) >> 2, p4 = lane & 3;
    const unsigned rdA = LDS_A + wm * 4096 + lane * 16;
    const unsigned rdB0 = LDS_B + 2048 * g + 512 * (2 * wc + (p4 >> 1)) + 64 * q4 + 32 * ((p4 ^ g) & 1);
    unsigned rdB[2][2];
#pragma unroll
    for (int u = 0; u < 2; ++u)
#pragma unroll
        for (int v = 0; v < 2; ++v) rdB[u][v] = rdB0 + 16 * (u ^ (p4 >> 1)) + 8 * (v ^ (p4 & 1));

    f32x4 acc[4][4];
#pragma unroll
    for (int i = 0; i < 4; ++i)
#pragma unroll
        for (int j = 0; j < 4; ++j) acc[i][j] = (f32x4){0.f, 0.f, 0.f, 0.f};
    struct Frags { s16x8 x[4], w[4]; };
    Frags f0, f1;
#pragma unroll
    for (int j = 0; j < 4; ++j) { f1.x[j] = (s16x8){0, 0, 0, 0, 0, 0, 0, 0}; f1.w[j] = f1.x[j]; }
    auto frag = [&](Frags& f, int slot, int j) {
        const s16x4 lo = __builtin_amdgcn_ds_read_tr16_b64_v4i16((LAS s16x4*)(lds + rdB[j >> 1][j & 1] + slot * B_STAGE));
        const s16x4 hi = __builtin_amdgcn_ds_read_tr16_b64_v4i16((LAS s16x4*)(lds + rdB[(j >> 1) ^ 1][j & 1] + slot * B_STAGE + 256));
        f.w[j] = __builtin_shufflevector(lo, hi, 0, 1, 2, 3, 4, 5, 6, 7);
        f.x[j] = *(const LAS s16x8*)(lds + rdA + slot * A_STAGE + j * 1024);
    };
    auto mm = [&](const Frags& f, int i, int j) { acc[i][j] = __builtin_amdgcn_mfma_f32_16x16x32_bf16(f.w[j], f.x[i], acc[i][j], 0, 0, 0); };
#define AG_NLEAD 1
    const bool follower = (tix >> 1) >= AG_NLEAD;
#define SB __builtin_amdgcn_sched_barrier(0)
    auto step = [&](auto INFLIGHT, auto SLOT, auto K, Frags& cur, Frags& prev) {
        constexpr int slot = decltype(SLOT)::value, k = decltype(K)::value;
        asm volatile("s_waitcnt vmcnt(%0) lgkmcnt(0)\n\ts_barrier" :: "n"(3 * decltype(INFLIGHT)::value) : "memory");
        if (AG_SCHED == 2) {
            frag(cur, slot, 0); SB; mm(prev, 0, 3); mm(prev, 1, 3); SB;
            frag(cur, slot, 1); SB; mm(prev, 2, 3); mm(prev, 3, 0); SB;
            frag(cur, slot, 2); SB; mm(prev, 3, 1); mm(prev, 3, 2); SB;
            frag(cur, slot, 3); SB; mm(prev, 3, 3); SB;
        } else if (AG_SCHED == 0) {
            frag(cur, slot, 0); frag(cur, slot, 1); frag(cur, slot, 2); frag(cur, slot, 3); SB; mm(cur, 0, 0);
        } else {
            frag(cur, slot, 0); frag(cur, slot, 1); frag(cur, slot, 2); SB; mm(cur, 0, 0); SB; frag(cur, slot, 3); SB;
        }
        constexpr int ns = (slot + NSTAGE - 1) % NSTAGE;
        if constexpr (AG_IPOS == 8 && k >= 0) { dma1<ns * A_STAGE, 0>(ldsWA, voffA[k < 0 ? 0 : k], gA); SB; }
        if constexpr (AG_IPOS == 7 && k >= 0) { if (wid < 4) issue(IC<k>{}, IC<ns>{}); SB; }
        if constexpr (AG_IPOS == 1 && k >= 0) { issue(IC<k>{}, IC<(slot + NSTAGE - 1) % NSTAGE>{}); SB; }
        if (AG_SCHED == 2) mm(cur, 0, 0);
        mm(cur, 0, 1); mm(cur, 1, 0); mm(cur, 1, 1); SB;
        if constexpr (AG_IPOS == 8 && k >= 0) { dma1<ns * A_STAGE, 1024>(ldsWA, voffA[k < 0 ? 0 : k], gA); SB; }
        mm(cur, 0, 2); mm(cur, 1, 2); mm(cur, 2, 0); mm(cur, 2, 1); mm(cur, 2, 2); SB;
        if (AG_SCHED != 2) { mm(cur, 0, 3); mm(cur, 1, 3); mm(cur, 2, 3); mm(cur, 3, 0); mm(cur, 3, 1); mm(cur, 3, 2); mm(cur, 3, 3); SB; }
        if (AG_XSLEEP) __builtin_amdgcn_s_sleep(AG_XSLEEP);
        if constexpr (AG_IPOS == 8 && k >= 0) { dma1<ns * B_STAGE, 0>(ldsWB, voffB[k < 0 ? 0 : k], gB); SB; }
        if constexpr (AG_IPOS == 7 && k >= 0) { if (wid >= 4) issue(IC<k>{}, IC<ns>{}); SB; }
        if constexpr (AG_IPOS == 3 && k >= 0) { issue(IC<k>{}, IC<(slot + NSTAGE - 1) % NSTAGE>{}); SB; }
    };

    asm volatile("" :: "s"(gA), "s"(gB), "v"(voffB[0]));
#define AG_NEED0 8
    if (wid == 0) wait_ready(AG_NEED0, 8);
    asm volatile("s_barrier" ::: "memory");
    STAMP(2);
    asm volatile("s_nop 4" ::: "memory");
    issue(IC<0>{}, IC<0>{}); issue(IC<1>{}, IC<1>{}); issue(IC<2>{}, IC<2>{});
    gA += 3 * 2 * A_STAGE; gB += 3 * BK * NL * 2;
    for (int s = 0; s < NSTEP - NSTAGE; s += NSTAGE) {
        if (wid == 0 && ready < NSTEP && s > 0) {
            const int need = s + 8 < NSTEP ? s + 8 : NSTEP;
            ready = ready_of(*(const volatile LAS u32x2*)(lds + LDS_X + 256 + lane * 16 + half * 8));
            if (ready < need) wait_ready(need, 64);
        }
        step(IC<2>{}, IC<0>{}, IC<0>{}, f0, f1);
        if (wid == 0 && ready < NSTEP) dma16_sys(lds0 + LDS_X + 256, fp + lane * 16);
        step(IC<2>{}, IC<1>{}, IC<1>{}, f1, f0);
        step(IC<2>{}, IC<2>{}, IC<2>{}, f0, f1);
        step(IC<2>{}, IC<3>{}, IC<3>{}, f1, f0);
        gA += NSTAGE * 2 * A_STAGE; gB += NSTAGE * BK * NL * 2;
    }
    step(IC<2>{}, IC<0>{}, IC<0>{}, f0, f1);
    step(IC<2>{}, IC<1>{}, IC<-1>{}, f1, f0);
    step(IC<1>{}, IC<2>{}, IC<-1>{}, f0, f1);
    step(IC<0>{}, IC<3>{}, IC<-1>{}, f1, f0);
    if (AG_SCHED == 2) { mm(f1, 0, 3); mm(f1, 1, 3); mm(f1, 2, 3); mm(f1, 3, 0); mm(f1, 3, 1); mm(f1, 3, 2); mm(f1, 3, 3); }
#undef SB
    STAMP(3);

    float mx = 0.f;
#pragma unroll
    for (int i = 0; i < 4; ++i)
#pragma unroll
        for (int j = 0; j < 4; ++j)
#pragma unroll
            for (int r = 0; r < 4; ++r) mx = fmaxf(mx, fabsf(acc[i][j][r]));
    {
        const unsigned m = wave_umax(__float_as_uint(mx));
        if (lane == 0) red[wid] = m;
    }
    asm volatile("s_waitcnt lgkmcnt(0)\n\ts_barrier" ::: "memory");
    const unsigned par = call & 1u;
    if (wid == 0) {
        const u32x4 a = *(const LAS u32x4*)red, b = *(const LAS u32x4*)(red + 4);
        const unsigned m = umax(umax(umax(a[0], a[1]), umax(a[2], a[3])), umax(umax(b[0], b[1]), umax(b[2], b[3])));
        if (lane == 0)
            __hip_atomic_store((unsigned long long*)(myws + WS_LMAX + par * (NWG * 8)) + bid, ((unsigned long long)call << 32) | m,
                               __ATOMIC_RELAXED, __HIP_MEMORY_SCOPE_AGENT);
    }
    STAMP(4);
    if (wid == 1 && srcr == P.rank && tix < NRED) {
        char* const dst = P.ws[lane & 7] + WS_GMAX + par * (NREP * 128) + (lane >> 3) * 128 + P.rank * 8;
        const char* gp = myws + WS_LMAX + par * (NWG * 8) + lane * 16;
        const unsigned long long t0 = __builtin_amdgcn_s_memrealtime();
        unsigned polls = 0;
        for (;;) {
            u32x4 a, b;
            load16x2_sys(gp, a, b);
            if (__all(a[1] == call && a[3] == call && b[1] == call && b[3] == call)) {
                const unsigned m = wave_umax(umax(umax(a[0], a[2]), umax(b[0], b[2])));
                __hip_atomic_store((unsigned long long*)dst, ((unsigned long long)call << 32) | m, __ATOMIC_RELAXED, __HIP_MEMORY_SCOPE_SYSTEM);
                break;
            }
            if (give_up(tmo, call, t0, polls)) break;
        }
    }
    if (wid == 2) {
        const char* gp = lane < 8 ? myws + WS_GMAX + par * (NREP * 128) + (tile & (NREP - 1)) * 128 + lane * 8 : (const char*)tmo;
        const unsigned long long t0 = __builtin_amdgcn_s_memrealtime();
        unsigned polls = 0, m = 0;
        bool dead = false;
        for (;;) {
            const u32x2 a = load8_sys(gp);
            if (__builtin_amdgcn_readlane(a[0], 8) == call) { dead = true; break; }
            if (__all(lane >= 8 || a[1] == call)) { m = wave_umax(lane < 8 ? a[0] : 0u); break; }
            if (give_up(tmo, call, t0, polls)) { dead = true; break; }
#define AG_FSLEEP 4
            __builtin_amdgcn_s_sleep(AG_FSLEEP);
        }
        if (lane == 0) { red[16] = m; red[17] = dead ? 1u : 0u; }
    }
    asm volatile("s_waitcnt lgkmcnt(0)\n\ts_barrier" ::: "memory");
    const unsigned gm = red[16], anydead = red[17];
    STAMP(5);

    const float amax = __uint_as_float(gm);
    const float inv = amax > 0.f ? 127.0f / amax : 0.f;
    char* op = P.out + (size_t)(srcr * MS + half * 256 + wm * 64 + (lane & 15)) * NL + n0 + wc * 64 + g * 16;
    const bool odd = g & 1;
#pragma unroll
    for (int i = 0; i < 4; ++i) {
        unsigned d[4];
#pragma unroll
        for (int j = 0; j < 4; ++j) {
            unsigned b[4];
#pragma unroll
            for (int r = 0; r < 4; ++r) {
                b[r] = __float_as_uint(__builtin_fmaf(acc[i][j][r], inv, 12582912.0f));
            }
            const unsigned lo = __builtin_amdgcn_perm(b[1], b[0], 0x0c0c0400u), hi = __builtin_amdgcn_perm(b[3], b[2], 0x04000c0cu);
            d[j] = lo | hi;
        }
        u32x4 o;
        o[0] = odd ? d[1] : d[0]; o[1] = odd ? d[0] : d[1]; o[2] = odd ? d[3] : d[2]; o[3] = odd ? d[2] : d[3];
        if (anydead) o = (u32x4){0x80808080u, 0x80808080u, 0x80808080u, 0x80808080u};
        store16_sys(op + (size_t)i * 16 * NL, o);
    }
    if (tid == 0) __hip_atomic_store(ctr, call, __ATOMIC_RELAXED, __HIP_MEMORY_SCOPE_AGENT);
    if (wid == 7 && __builtin_amdgcn_readfirstlane(xold) != call) asm volatile("buffer_inv sc0 sc1" ::: "memory");
    STAMP(6);
}

}

extern "C" void kernel_launch(int rank, int nranks, ncclComm_t comm,
                              hipStream_t stream,
                              void* const* d_in, const int* in_sizes, int n_in,
                              void* d_out, int out_size,
                              const int* in_dtypes, int out_dtype,
                              void* const* d_out_all,
                              void* const* d_ws_all, size_t ws_bytes) {
    (void)comm; (void)in_dtypes; (void)out_dtype; (void)d_out_all;
    if (nranks != NR || n_in < 2 || rank < 0 || rank >= NR || in_sizes[0] != MS * KD || in_sizes[1] != KD * NL ||
        out_size != NR * MS * NL || ws_bytes < WS_END) {
        fprintf(stderr, "kernel_launch: unexpected problem shape\n");
        return;
    }
    Params P;
    P.x = (const char*)d_in[0];
    P.w = (const char*)d_in[1];
    P.out = (char*)d_out;
    for (int p = 0; p < NR; ++p) P.ws[p] = (char*)d_ws_all[p];
    P.myws = P.ws[rank];
    P.rank = rank;
    P.map = AG_MAP(rank);
    HIP_CHECK(hipFuncSetAttribute((const void*)ag_gemm_q8, hipFuncAttributeMaxDynamicSharedMemorySize, LDS_BYTES));
    hipLaunchKernelGGL(ag_gemm_q8, dim3(NWG), dim3(512), LDS_BYTES, stream, P);
    HIP_CHECK(hipGetLastError());
}
```

```cpp
#include <hip/hip_runtime.h>
#include <rccl/rccl.h>
#include <cstdio>

#define HIP_CHECK(expr) do { hipError_t _e = (expr); if (_e != hipSuccess) { \
    fprintf(stderr, "HIP error %s at %s:%d\n", hipGetErrorString(_e), __FILE__, __LINE__); \
    return; } } while (0)

namespace {

#define AG_MAP(rank) ((((0x4Du >> (rank)) & 1u) != 0u) ? 0x75416320u : 0x63207541u)
#define AG_SRC(g, map) ((int)(((map) >> (4 * ((g) & 7))) & 7u))

#define LAS __attribute__((address_space(3)))
typedef short s16x4 __attribute__((ext_vector_type(4)));
typedef short s16x8 __attribute__((ext_vector_type(8)));
typedef float f32x4 __attribute__((ext_vector_type(4)));
typedef unsigned u32x4 __attribute__((ext_vector_type(4)));
typedef unsigned u32x2 __attribute__((ext_vector_type(2)));

constexpr int NR = 8;
constexpr int MS = 512;
constexpr int KD = 4096;
constexpr int NL = 2048;
constexpr int NWG = 256;
constexpr int BK = 32, NSTEP = KD / BK;
constexpr int NSTAGE = 4;
constexpr int A_STAGE = 256 * BK * 2;
constexpr int B_STAGE = BK * 128 * 2;
constexpr int LDS_A = 0, LDS_B = NSTAGE * A_STAGE, LDS_X = LDS_B + NSTAGE * B_STAGE;
constexpr int LDS_BYTES = LDS_X + 256 + 1024;
constexpr int NRED = 2;
constexpr int NREP = 8;

constexpr size_t WS_XS = 0;
constexpr size_t WS_FLAG = (size_t)MS * KD * 2;
constexpr size_t WS_LMAX = WS_FLAG + 65536;
constexpr size_t WS_GMAX = WS_LMAX + 65536;
constexpr size_t WS_CTR = WS_GMAX + 65536;
constexpr size_t WS_TMO = WS_CTR + 4096;
constexpr size_t WS_XCC = WS_TMO + 4096;
constexpr size_t WS_CLAIM = WS_XCC + 4096;
constexpr size_t WS_DBG = WS_CLAIM + 16384;
constexpr size_t WS_END = WS_DBG + 65536;
constexpr unsigned long long DEADLINE = 50000000ull;

template <int N> struct IC { static constexpr int value = N; };

struct Params {
    const char* x;
    const char* w;
    char* out;
    char* myws;
    char* ws[NR];
    int rank;
    unsigned map;
};

__device__ __forceinline__ u32x2 load8_sys(const char* p) {
    u32x2 v;
    asm volatile("global_load_dwordx2 %0, %1, off sc0 sc1\n\ts_waitcnt vmcnt(0)" : "=&v"(v) : "v"(p) : "memory");
    return v;
}
__device__ __forceinline__ void load16x2_sys(const char* p, u32x4& a, u32x4& b) {
    asm volatile("global_load_dwordx4 %0, %2, off sc0 sc1\n\tglobal_load_dwordx4 %1, %2, off offset:1024 sc0 sc1\n\ts_waitcnt vmcnt(0)"
                 : "=&v"(a), "=&v"(b) : "v"(p) : "memory");
}
__device__ __forceinline__ void dma16_sys(unsigned lds_addr, const char* p) {
    unsigned keep;
    asm volatile("s_mov_b32 %0, m0\n\ts_mov_b32 m0, %1\n\ts_nop 0\n\tglobal_load_lds_dwordx4 %2, off sc0 sc1\n\ts_mov_b32 m0, %0"
                 : "=&s"(keep) : "s"(lds_addr), "v"(p) : "memory");
}
template <int SLOT>
__device__ __forceinline__ void stage_dma(unsigned ldsA, unsigned ldsB, unsigned va, const char* ga, unsigned vb, const char* gb) {
    asm volatile("s_add_i32 m0, %0, %6\n\ts_nop 0\n\t"
                 "global_load_lds_dwordx4 %2, %3\n\t"
                 "global_load_lds_dwordx4 %2, %3 offset:1024\n\t"
                 "s_add_i32 m0, %1, %7\n\ts_nop 0\n\t"
                 "global_load_lds_dwordx4 %4, %5"
                 :: "s"(ldsA), "s"(ldsB), "v"(va), "s"(ga), "v"(vb), "s"(gb), "n"(SLOT * A_STAGE), "n"(SLOT * B_STAGE) : "memory", "scc");
}
__device__ __forceinline__ void store16_sys(char* p, u32x4 v) {
    asm volatile("global_store_dwordx4 %0, %1, off sc0 sc1\n\ts_nop 1" :: "v"(p), "v"(v) : "memory");
}
__device__ __forceinline__ unsigned umax(unsigned a, unsigned b) { return a > b ? a : b; }
__device__ __forceinline__ unsigned wave_umax(unsigned v) {
    v = umax(v, (unsigned)__builtin_amdgcn_update_dpp(0, (int)v, 0xB1, 0xf, 0xf, true));
    v = umax(v, (unsigned)__builtin_amdgcn_update_dpp(0, (int)v, 0x4E, 0xf, 0xf, true));
    v = umax(v, (unsigned)__builtin_amdgcn_update_dpp(0, (int)v, 0x141, 0xf, 0xf, true));
    v = umax(v, (unsigned)__builtin_amdgcn_update_dpp(0, (int)v, 0x140, 0xf, 0xf, true));
    const unsigned a = __builtin_amdgcn_readlane(v, 0), b = __builtin_amdgcn_readlane(v, 16), c = __builtin_amdgcn_readlane(v, 32), d = __builtin_amdgcn_readlane(v, 48);
    return umax(umax(a, b), umax(c, d));
}
__device__ __forceinline__ bool give_up(unsigned* tmo, unsigned call, unsigned long long t0, unsigned& polls) {
    if ((++polls & 15u) != 0u) return false;
    const unsigned t = __builtin_amdgcn_readfirstlane(__hip_atomic_load(tmo, __ATOMIC_RELAXED, __HIP_MEMORY_SCOPE_AGENT));
    if (t == call || __builtin_amdgcn_s_memrealtime() - t0 > DEADLINE) {
        __hip_atomic_store(tmo, call, __ATOMIC_RELAXED, __HIP_MEMORY_SCOPE_AGENT);
        return true;
    }
    return false;
}

#define STAMP(i) do { } while (0)

__global__ __launch_bounds__(512, 2) void ag_gemm_q8(Params P) {
    extern __shared__ __attribute__((aligned(16))) char smem[];
    LAS char* lds = (LAS char*)smem;
    const int tid = threadIdx.x, lane = tid & 63, wid = __builtin_amdgcn_readfirstlane(tid >> 6);
    const int bid = blockIdx.x;
    char* const myws = P.myws;
    asm volatile("" :: "s"(P.x), "s"(P.w), "s"(P.out), "s"(P.myws), "s"(P.rank), "s"(P.map), "s"(P.ws[0]), "s"(P.ws[1]), "s"(P.ws[2]),
                 "s"(P.ws[3]), "s"(P.ws[4]), "s"(P.ws[5]), "s"(P.ws[6]), "s"(P.ws[7]));
    unsigned* const tmo = (unsigned*)(myws + WS_TMO);
    unsigned* const ctr = (unsigned*)(myws + WS_CTR) + bid;
    LAS unsigned* const red = (LAS unsigned*)(lds + LDS_X);
    STAMP(0);

    unsigned call;
    unsigned xold = 0;
    if (wid < 4) {
        const int kt = bid >> 2, q = bid & 3;
        const int r0 = tid >> 3, c = tid & 7;
        const char* src = P.x + (size_t)(q * 128 + r0) * (KD * 2) + kt * 128 + c * 16;
        u32x4 v[4];
#pragma unroll
        for (int i = 0; i < 4; ++i) v[i] = *(const u32x4*)(src + (size_t)i * 32 * (KD * 2));
        call = __hip_atomic_load(ctr, __ATOMIC_RELAXED, __HIP_MEMORY_SCOPE_AGENT) + 1u;
        const int step = 2 * kt + (c >> 2), rt = (q & 1) * 128 + r0;
        char* dst = myws + WS_XS + (size_t)(step * 2 + (q >> 1)) * A_STAGE + (rt >> 4) * 1024 + ((c & 3) * 16 + (rt & 15)) * 16;
        asm volatile("" : "+v"(v[0]), "+v"(v[1]), "+v"(v[2]), "+v"(v[3]), "+v"(call));
#pragma unroll
        for (int i = 0; i < 4; ++i) store16_sys(dst + i * 2048, v[i]);
        asm volatile("s_waitcnt vmcnt(0)\n\ts_barrier" ::: "memory");
        if (tid == 64) {
            const unsigned foff = (unsigned)WS_FLAG + (unsigned)(P.rank * NWG + bid) * 4u;
#pragma unroll
            for (int p = 0; p < NR; ++p) asm volatile("global_store_dword %0, %1, %2 sc0 sc1" :: "v"(foff), "v"(call), "s"(P.ws[p]) : "memory");
        }
    } else {
        call = __hip_atomic_load(ctr, __ATOMIC_RELAXED, __HIP_MEMORY_SCOPE_AGENT) + 1u;
        asm volatile("" : "+v"(call));
        const unsigned xcc = (unsigned)__builtin_amdgcn_s_getreg((3 << 11) | 20);
        if (wid == 4) {
            const unsigned ucall = __builtin_amdgcn_readfirstlane(call);
            unsigned t = ((unsigned)bid & ~7u) | (xcc & 7u);
#pragma unroll 1
            for (int i = 0; i < NWG; ++i) {
                unsigned old = 0;
                if (lane == 0) old = __hip_atomic_exchange((unsigned*)(myws + WS_CLAIM) + t * 16, ucall, __ATOMIC_RELAXED, __HIP_MEMORY_SCOPE_AGENT);
                if (__builtin_amdgcn_readfirstlane(old) != ucall) break;
                t = (t + 1) & (NWG - 1);
            }
            if (lane == 0) red[32] = t;
        }
        asm volatile("s_waitcnt lgkmcnt(0)\n\ts_barrier" ::: "memory");
        if (wid == 7 && lane == 0)
            xold = __hip_atomic_exchange((unsigned*)(myws + WS_XCC) + xcc * 32, call, __ATOMIC_RELAXED, __HIP_MEMORY_SCOPE_AGENT);
    }
    call = __builtin_amdgcn_readfirstlane(call);
    STAMP(1);

    const int tile = __builtin_amdgcn_readfirstlane(*(const volatile LAS unsigned*)(red + 32));
    const int srcr = AG_SRC(tile, P.map);
    const int tix = tile >> 3, half = tix & 1, n0 = (tix >> 1) * 128;

    const unsigned lds0 = (unsigned)(size_t)lds;
    int ready = 0;
    const char* const fp = myws + WS_FLAG + (size_t)srcr * (NWG * 4);
    auto ready_of = [&](u32x2 f) {
        const unsigned long long miss = ~__ballot(f[0] >= call && f[1] >= call);
        return miss ? 2 * (int)__builtin_ctzll(miss) : NSTEP;
    };
    auto wait_ready = [&](int need, int tiles) {
        const unsigned long long t0 = __builtin_amdgcn_s_memrealtime();
        unsigned polls = 0;
        for (;;) {
            ready = ready_of(load8_sys(fp + (lane & (tiles - 1)) * 16 + half * 8));
            if (ready > 2 * tiles) ready = 2 * tiles;
            if (ready >= need) break;
            if (give_up(tmo, call, t0, polls)) { ready = NSTEP; break; }
        }
    };

    const char* gA = P.ws[srcr] + WS_XS + (size_t)half * A_STAGE + wid * 2048;
    const char* gB = P.w + n0 * 2;
    unsigned voffA[NSTAGE], voffB[NSTAGE];
    {
        const int row = 8 * (wid >> 1) + ((lane >> 2) & 7), chh = 2 * (wid & 1) + (lane >> 5);
        const int ch = chh * 4 + ((lane & 3) ^ ((row >> 2) & 3) ^ (chh & 1));
#pragma unroll
        for (int k = 0; k < NSTAGE; ++k) {
            voffA[k] = (unsigned)(lane * 16 + k * 2 * A_STAGE);
            voffB[k] = (unsigned)(row * (NL * 2) + ch * 16 + k * BK * NL * 2);
        }
    }
    const unsigned ldsWA = lds0 + LDS_A + wid * 2048, ldsWB = lds0 + LDS_B + wid * 1024;
    auto issue = [&](auto K, auto SLOT) {
        stage_dma<decltype(SLOT)::value>(ldsWA, ldsWB, voffA[decltype(K)::value], gA, voffB[decltype(K)::value], gB);
    };

    const int wm = wid >> 1, wc = wid & 1;
    const int g = lane >> 4, q4 = (lane & 15) >> 2, p4 = lane & 3;
    const unsigned rdA = LDS_A + wm * 4096 + lane * 16;
    const unsigned rdB0 = LDS_B + 2048 * g + 512 * (2 * wc + (p4 >> 1)) + 64 * q4 + 32 * ((p4 ^ g) & 1);
    unsigned rdB[2][2];
#pragma unroll
    for (int u = 0; u < 2; ++u)
#pragma unroll
        for (int v = 0; v < 2; ++v) rdB[u][v] = rdB0 + 16 * (u ^ (p4 >> 1)) + 8 * (v ^ (p4 & 1));

    f32x4 acc[4][4];
#pragma unroll
    for (int i = 0; i < 4; ++i)
#pragma unroll
        for (int j = 0; j < 4; ++j) acc[i][j] = (f32x4){0.f, 0.f, 0.f, 0.f};
    struct Frags { s16x8 x[4], w[4]; };
    Frags f0, f1;
#pragma unroll
    for (int j = 0; j < 4; ++j) { f1.x[j] = (s16x8){0, 0, 0, 0, 0, 0, 0, 0}; f1.w[j] = f1.x[j]; }
    auto frag = [&](Frags& f, int slot, int j) {
        const s16x4 lo = __builtin_amdgcn_ds_read_tr16_b64_v4i16((LAS s16x4*)(lds + rdB[j >> 1][j & 1] + slot * B_STAGE));
        const s16x4 hi = __builtin_amdgcn_ds_read_tr16_b64_v4i16((LAS s16x4*)(lds + rdB[(j >> 1) ^ 1][j & 1] + slot * B_STAGE + 256));
        f.w[j] = __builtin_shufflevector(lo, hi, 0, 1, 2, 3, 4, 5, 6, 7);
        f.x[j] = *(const LAS s16x8*)(lds + rdA + slot * A_STAGE + j * 1024);
    };
    auto mm = [&](const Frags& f, int i, int j) { acc[i][j] = __builtin_amdgcn_mfma_f32_16x16x32_bf16(f.w[j], f.x[i], acc[i][j], 0, 0, 0); };
#define SB __builtin_amdgcn_sched_barrier(0)
    auto step = [&](auto INFLIGHT, auto SLOT, auto K, Frags& cur, Frags& prev) {
        constexpr int slot = decltype(SLOT)::value, k = decltype(K)::value, ns = (slot + NSTAGE - 1) % NSTAGE;
        asm volatile("s_waitcnt vmcnt(%0) lgkmcnt(0)\n\ts_barrier" :: "n"(3 * decltype(INFLIGHT)::value) : "memory");
        frag(cur, slot, 0); SB; mm(prev, 0, 3); mm(prev, 1, 3); SB;
        frag(cur, slot, 1); SB; mm(prev, 2, 3); mm(prev, 3, 0); SB;
        frag(cur, slot, 2); SB; mm(prev, 3, 1); mm(prev, 3, 2); SB;
        frag(cur, slot, 3); SB; mm(prev, 3, 3); SB;
        if constexpr (k >= 0) { if (wid < 4) issue(IC<k>{}, IC<ns>{}); SB; }
        mm(cur, 0, 0); mm(cur, 0, 1); mm(cur, 1, 0); mm(cur, 1, 1); SB;
        mm(cur, 0, 2); mm(cur, 1, 2); mm(cur, 2, 0); mm(cur, 2, 1); mm(cur, 2, 2); SB;
        if constexpr (k >= 0) { if (wid >= 4) issue(IC<k>{}, IC<ns>{}); SB; }
    };

    asm volatile("" :: "s"(gA), "s"(gB), "v"(voffB[0]));
    if (wid == 0) wait_ready(8, 8);
    asm volatile("s_barrier" ::: "memory");
    STAMP(2);
    asm volatile("s_nop 4" ::: "memory");
    issue(IC<0>{}, IC<0>{}); issue(IC<1>{}, IC<1>{}); issue(IC<2>{}, IC<2>{});
    gA += 3 * 2 * A_STAGE; gB += 3 * BK * NL * 2;
    for (int s = 0; s < NSTEP - NSTAGE; s += NSTAGE) {
        if (wid == 0 && ready < NSTEP && s > 0) {
            const int need = s + 8 < NSTEP ? s + 8 : NSTEP;
            ready = ready_of(*(const volatile LAS u32x2*)(lds + LDS_X + 256 + lane * 16 + half * 8));
            if (ready < need) wait_ready(need, 64);
        }
        step(IC<2>{}, IC<0>{}, IC<0>{}, f0, f1);
        if (wid == 0 && ready < NSTEP) dma16_sys(lds0 + LDS_X + 256, fp + lane * 16);
        step(IC<2>{}, IC<1>{}, IC<1>{}, f1, f0);
        step(IC<2>{}, IC<2>{}, IC<2>{}, f0, f1);
        step(IC<2>{}, IC<3>{}, IC<3>{}, f1, f0);
        gA += NSTAGE * 2 * A_STAGE; gB += NSTAGE * BK * NL * 2;
    }
    step(IC<2>{}, IC<0>{}, IC<0>{}, f0, f1);
    step(IC<2>{}, IC<1>{}, IC<-1>{}, f1, f0);
    step(IC<1>{}, IC<2>{}, IC<-1>{}, f0, f1);
    step(IC<0>{}, IC<3>{}, IC<-1>{}, f1, f0);
    mm(f1, 0, 3); mm(f1, 1, 3); mm(f1, 2, 3); mm(f1, 3, 0); mm(f1, 3, 1); mm(f1, 3, 2); mm(f1, 3, 3);
#undef SB
    STAMP(3);

    float mx = 0.f;
#pragma unroll
    for (int i = 0; i < 4; ++i)
#pragma unroll
        for (int j = 0; j < 4; ++j)
#pragma unroll
            for (int r = 0; r < 4; ++r) mx = fmaxf(mx, fabsf(acc[i][j][r]));
    {
        const unsigned m = wave_umax(__float_as_uint(mx));
        if (lane == 0) red[wid] = m;
    }
    asm volatile("s_waitcnt lgkmcnt(0)\n\ts_barrier" ::: "memory");
    const unsigned par = call & 1u;
    if (wid == 0) {
        const u32x4 a = *(const LAS u32x4*)red, b = *(const LAS u32x4*)(red + 4);
        const unsigned m = umax(umax(umax(a[0], a[1]), umax(a[2], a[3])), umax(umax(b[0], b[1]), umax(b[2], b[3])));
        if (lane == 0)
            __hip_atomic_store((unsigned long long*)(myws + WS_LMAX + par * (NWG * 8)) + bid, ((unsigned long long)call << 32) | m,
                               __ATOMIC_RELAXED, __HIP_MEMORY_SCOPE_AGENT);
    }
    STAMP(4);
    f32x4 qv[4][4];
    const bool hi8 = (lane & 8) != 0;
    {
        f32x4 own[2][4], oth[2][4];
        LAS char* xw = lds + wid * 8192 + lane * 16;
        if (wc == 0) {
#pragma unroll
            for (int i = 0; i < 2; ++i)
#pragma unroll
                for (int j = 0; j < 4; ++j) { *(LAS f32x4*)(xw + (i * 4 + j) * 1024) = acc[2 + i][j]; own[i][j] = acc[i][j]; }
        } else {
#pragma unroll
            for (int i = 0; i < 2; ++i)
#pragma unroll
                for (int j = 0; j < 4; ++j) { *(LAS f32x4*)(xw + (i * 4 + j) * 1024) = acc[i][j]; own[i][j] = acc[2 + i][j]; }
        }
        asm volatile("s_waitcnt lgkmcnt(0)\n\ts_barrier" ::: "memory");
        const LAS char* xr = lds + (wid ^ 1) * 8192 + (lane ^ 8) * 16;
#pragma unroll
        for (int i = 0; i < 2; ++i)
#pragma unroll
            for (int j = 0; j < 4; ++j) oth[i][j] = *(const LAS f32x4*)(xr + (i * 4 + j) * 1024);
        const bool odd = g & 1;
#pragma unroll
        for (int i = 0; i < 2; ++i)
#pragma unroll
            for (int j = 0; j < 4; ++j) {
                const f32x4 a0 = own[i][j], a1 = own[i][j ^ 1], b0 = oth[i][j], b1 = oth[i][j ^ 1];
#pragma unroll
                for (int r = 0; r < 4; ++r) {
                    const float a = odd ? a1[r] : a0[r], b = odd ? b1[r] : b0[r];
                    qv[2 * i][j][r] = hi8 ? b : a;
                    qv[2 * i + 1][j][r] = hi8 ? a : b;
                }
                asm volatile("" : "+v"(qv[2 * i][j]), "+v"(qv[2 * i + 1][j]));
            }
    }
    char* op = P.out + (size_t)(srcr * MS + half * 256 + wm * 64 + wc * 32 + (lane & 7)) * NL + n0 + g * 16;
    const int col0 = hi8 ? 64 - wc * 64 : wc * 64, col1 = 64 - col0;
    asm volatile("" : "+v"(op));
    if (wid == 1 && srcr == P.rank && tix < NRED) {
        char* const dst = P.ws[lane & 7] + WS_GMAX + par * (NREP * 128) + (lane >> 3) * 128 + P.rank * 8;
        const char* gp = myws + WS_LMAX + par * (NWG * 8) + lane * 16;
        const unsigned long long t0 = __builtin_amdgcn_s_memrealtime();
        unsigned polls = 0;
        for (;;) {
            u32x4 a, b;
            load16x2_sys(gp, a, b);
            if (__all(a[1] == call && a[3] == call && b[1] == call && b[3] == call)) {
                const unsigned m = wave_umax(umax(umax(a[0], a[2]), umax(b[0], b[2])));
                __hip_atomic_store((unsigned long long*)dst, ((unsigned long long)call << 32) | m, __ATOMIC_RELAXED, __HIP_MEMORY_SCOPE_SYSTEM);
                break;
            }
            if (give_up(tmo, call, t0, polls)) break;
        }
    }
    if (wid == 2) {
        const char* gp = lane < 8 ? myws + WS_GMAX + par * (NREP * 128) + (tile & (NREP - 1)) * 128 + lane * 8 : (const char*)tmo;
        const unsigned long long t0 = __builtin_amdgcn_s_memrealtime();
        unsigned polls = 0, m = 0;
        bool dead = false;
        for (;;) {
            const u32x2 a = load8_sys(gp);
            if (__builtin_amdgcn_readlane(a[0], 8) == call) { dead = true; break; }
            if (__all(lane >= 8 || a[1] == call)) { m = a[0]; break; }
            if (give_up(tmo, call, t0, polls)) { dead = true; break; }
            __builtin_amdgcn_s_sleep(2);
        }
        m = umax(m, (unsigned)__builtin_amdgcn_update_dpp(0, (int)m, 0xB1, 0xf, 0xf, true));
        m = umax(m, (unsigned)__builtin_amdgcn_update_dpp(0, (int)m, 0x4E, 0xf, 0xf, true));
        m = umax(m, (unsigned)__builtin_amdgcn_update_dpp(0, (int)m, 0x141, 0xf, 0xf, true));
        const float amax = __uint_as_float(m);
        const float inv = amax > 0.f ? 127.0f / amax : 0.f;
        if (lane == 0) { red[16] = __float_as_uint(inv); red[17] = dead ? 1u : 0u; }
    }
    asm volatile("s_waitcnt lgkmcnt(0)\n\ts_barrier" ::: "memory");
    const float inv = __uint_as_float(red[16]);
    const unsigned anydead = red[17];
    STAMP(5);

#pragma unroll
    for (int i = 0; i < 4; ++i) {
        u32x4 o;
#pragma unroll
        for (int j = 0; j < 4; ++j) {
            unsigned b[4];
#pragma unroll
            for (int r = 0; r < 4; ++r) {
                b[r] = __float_as_uint(__builtin_fmaf(qv[i][j][r], inv, 12582912.0f));
            }
            const unsigned lo = __builtin_amdgcn_perm(b[1], b[0], 0x0c0c0400u), hi = __builtin_amdgcn_perm(b[3], b[2], 0x04000c0cu);
            o[j] = lo | hi;
        }
        if (anydead) o = (u32x4){0x80808080u, 0x80808080u, 0x80808080u, 0x80808080u};
        store16_sys(op + (size_t)(i * 8) * NL + ((i & 1) ? col1 : col0), o);
        __builtin_amdgcn_sched_barrier(0);
    }
    if (tid == 0) __hip_atomic_store(ctr, call, __ATOMIC_RELAXED, __HIP_MEMORY_SCOPE_AGENT);
    if (wid == 7 && __builtin_amdgcn_readfirstlane(xold) != call) asm volatile("buffer_inv sc0 sc1" ::: "memory");
    STAMP(6);
}

}

extern "C" void kernel_launch(int rank, int nranks, ncclComm_t comm,
                              hipStream_t stream,
                              void* const* d_in, const int* in_sizes, int n_in,
                              void* d_out, int out_size,
                              const int* in_dtypes, int out_dtype,
                              void* const* d_out_all,
                              void* const* d_ws_all, size_t ws_bytes) {
    (void)comm; (void)in_dtypes; (void)out_dtype; (void)d_out_all;
    if (nranks != NR || n_in < 2 || rank < 0 || rank >= NR || in_sizes[0] != MS * KD || in_sizes[1] != KD * NL ||
        out_size != NR * MS * NL || ws_bytes < WS_END) {
        fprintf(stderr, "kernel_launch: unexpected problem shape\n");
        return;
    }
    Params P;
    P.x = (const char*)d_in[0];
    P.w = (const char*)d_in[1];
    P.out = (char*)d_out;
    for (int p = 0; p < NR; ++p) P.ws[p] = (char*)d_ws_all[p];
    P.myws = P.ws[rank];
    P.rank = rank;
    P.map = AG_MAP(rank);
    HIP_CHECK(hipFuncSetAttribute((const void*)ag_gemm_q8, hipFuncAttributeMaxDynamicSharedMemorySize, LDS_BYTES));
    hipLaunchKernelGGL(ag_gemm_q8, dim3(NWG), dim3(512), LDS_BYTES, stream, P);
    HIP_CHECK(hipGetLastError());
}
```

```cpp
#include <hip/hip_runtime.h>
#include <rccl/rccl.h>
#include <cstdio>

#define HIP_CHECK(expr) do { hipError_t _e = (expr); if (_e != hipSuccess) { \
    fprintf(stderr, "HIP error %s at %s:%d\n", hipGetErrorString(_e), __FILE__, __LINE__); \
    return; } } while (0)

namespace {

#define AG_MAP(rank) ((((0x4Du >> (rank)) & 1u) != 0u) ? 0x75416320u : 0x63207541u)
#define AG_SRC(g, map) ((int)(((map) >> (4 * ((g) & 7))) & 7u))

#define LAS __attribute__((address_space(3)))
typedef short s16x4 __attribute__((ext_vector_type(4)));
typedef short s16x8 __attribute__((ext_vector_type(8)));
typedef float f32x4 __attribute__((ext_vector_type(4)));
typedef unsigned u32x4 __attribute__((ext_vector_type(4)));
typedef unsigned u32x2 __attribute__((ext_vector_type(2)));

constexpr int NR = 8;
constexpr int MS = 512;
constexpr int KD = 4096;
constexpr int NL = 2048;
constexpr int NWG = 256;
constexpr int BK = 32, NSTEP = KD / BK;
constexpr int NSTAGE = 4;
constexpr int A_STAGE = 256 * BK * 2;
constexpr int B_STAGE = BK * 128 * 2;
constexpr int LDS_A = 0, LDS_B = NSTAGE * A_STAGE, LDS_X = LDS_B + NSTAGE * B_STAGE;
constexpr int LDS_BYTES = LDS_X + 256 + 1024;
constexpr int NRED = 2;
constexpr int NREP = 8;

constexpr size_t WS_XS = 0;
constexpr size_t WS_FLAG = (size_t)MS * KD * 2;
constexpr size_t WS_LMAX = WS_FLAG + 65536;
constexpr size_t WS_GMAX = WS_LMAX + 65536;
constexpr size_t WS_CTR = WS_GMAX + 65536;
constexpr size_t WS_TMO = WS_CTR + 4096;
constexpr size_t WS_XCC = WS_TMO + 4096;
constexpr size_t WS_CLAIM = WS_XCC + 4096;
constexpr size_t WS_DBG = WS_CLAIM + 16384;
constexpr size_t WS_END = WS_DBG + 65536;
constexpr unsigned long long DEADLINE = 50000000ull;

template <int N> struct IC { static constexpr int value = N; };

struct Params {
    const char* x;
    const char* w;
    char* out;
    char* myws;
    char* ws[NR];
    int rank;
    unsigned map;
};

__device__ __forceinline__ u32x2 load8_sys(const char* p) {
    u32x2 v;
    asm volatile("global_load_dwordx2 %0, %1, off sc0 sc1\n\ts_waitcnt vmcnt(0)" : "=&v"(v) : "v"(p) : "memory");
    return v;
}
__device__ __forceinline__ u32x4 load16_sys(const char* p) {
    u32x4 v;
    asm volatile("global_load_dwordx4 %0, %1, off sc0 sc1\n\ts_waitcnt vmcnt(0)" : "=&v"(v) : "v"(p) : "memory");
    return v;
}
__device__ __forceinline__ void load16x2_sys(const char* p, u32x4& a, u32x4& b) {
    asm volatile("global_load_dwordx4 %0, %2, off sc0 sc1\n\tglobal_load_dwordx4 %1, %2, off offset:1024 sc0 sc1\n\ts_waitcnt vmcnt(0)"
                 : "=&v"(a), "=&v"(b) : "v"(p) : "memory");
}
__device__ __forceinline__ void dma16_sys(unsigned lds_addr, const char* p) {
    unsigned keep;
    asm volatile("s_mov_b32 %0, m0\n\ts_mov_b32 m0, %1\n\ts_nop 0\n\tglobal_load_lds_dwordx4 %2, off sc0 sc1\n\ts_mov_b32 m0, %0"
                 : "=&s"(keep) : "s"(lds_addr), "v"(p) : "memory");
}
template <int SLOT>
__device__ __forceinline__ void stage_dma(unsigned ldsA, unsigned ldsB, unsigned va, const char* ga, unsigned vb, const char* gb) {
    asm volatile("s_add_i32 m0, %0, %6\n\ts_nop 0\n\t"
                 "global_load_lds_dwordx4 %2, %3\n\t"
                 "global_load_lds_dwordx4 %2, %3 offset:1024\n\t"
                 "s_add_i32 m0, %1, %7\n\ts_nop 0\n\t"
                 "global_load_lds_dwordx4 %4, %5"
                 :: "s"(ldsA), "s"(ldsB), "v"(va), "s"(ga), "v"(vb), "s"(gb), "n"(SLOT * A_STAGE), "n"(SLOT * B_STAGE) : "memory", "scc");
}
__device__ __forceinline__ void store16_sys(char* p, u32x4 v) {
    asm volatile("global_store_dwordx4 %0, %1, off sc0 sc1\n\ts_nop 1" :: "v"(p), "v"(v) : "memory");
}
__device__ __forceinline__ unsigned umax(unsigned a, unsigned b) { return a > b ? a : b; }
__device__ __forceinline__ unsigned wave_umax(unsigned v) {
    v = umax(v, (unsigned)__builtin_amdgcn_update_dpp(0, (int)v, 0xB1, 0xf, 0xf, true));
    v = umax(v, (unsigned)__builtin_amdgcn_update_dpp(0, (int)v, 0x4E, 0xf, 0xf, true));
    v = umax(v, (unsigned)__builtin_amdgcn_update_dpp(0, (int)v, 0x141, 0xf, 0xf, true));
    v = umax(v, (unsigned)__builtin_amdgcn_update_dpp(0, (int)v, 0x140, 0xf, 0xf, true));
    const unsigned a = __builtin_amdgcn_readlane(v, 0), b = __builtin_amdgcn_readlane(v, 16), c = __builtin_amdgcn_readlane(v, 32), d = __builtin_amdgcn_readlane(v, 48);
    return umax(umax(a, b), umax(c, d));
}
__device__ __forceinline__ bool give_up(unsigned* tmo, unsigned call, unsigned long long t0, unsigned& polls) {
    if ((++polls & 15u) != 0u) return false;
    const unsigned t = __builtin_amdgcn_readfirstlane(__hip_atomic_load(tmo, __ATOMIC_RELAXED, __HIP_MEMORY_SCOPE_AGENT));
    if (t == call || __builtin_amdgcn_s_memrealtime() - t0 > DEADLINE) {
        __hip_atomic_store(tmo, call, __ATOMIC_RELAXED, __HIP_MEMORY_SCOPE_AGENT);
        return true;
    }
    return false;
}

#define STAMP(i) do { } while (0)

__global__ __launch_bounds__(512, 2) void ag_gemm_q8(Params P) {
    extern __shared__ __attribute__((aligned(16))) char smem[];
    LAS char* lds = (LAS char*)smem;
    const int tid = threadIdx.x, lane = tid & 63, wid = __builtin_amdgcn_readfirstlane(tid >> 6);
    const int bid = blockIdx.x;
    char* const myws = P.myws;
    asm volatile("" :: "s"(P.x), "s"(P.w), "s"(P.out), "s"(P.myws), "s"(P.rank), "s"(P.map), "s"(P.ws[0]), "s"(P.ws[1]), "s"(P.ws[2]),
                 "s"(P.ws[3]), "s"(P.ws[4]), "s"(P.ws[5]), "s"(P.ws[6]), "s"(P.ws[7]));
    unsigned* const tmo = (unsigned*)(myws + WS_TMO);
    unsigned* const ctr = (unsigned*)(myws + WS_CTR) + bid;
    LAS unsigned* const red = (LAS unsigned*)(lds + LDS_X);
    STAMP(0);

    unsigned call;
    unsigned xold = 0;
    if (wid < 4) {
        const int kp = bid >> 3, e = bid & 7;
        const int r0 = tid >> 3, c = tid & 7;
        const char* src = P.x + (size_t)(e * 64 + r0) * (KD * 2) + kp * 256 + c * 16;
        if (kp >= 2) __builtin_amdgcn_s_sleep(5);
        u32x4 v[4];
#pragma unroll
        for (int i = 0; i < 4; ++i) v[i] = *(const u32x4*)(src + (size_t)(i >> 1) * 32 * (KD * 2) + (i & 1) * 128);
        call = __hip_atomic_load(ctr, __ATOMIC_RELAXED, __HIP_MEMORY_SCOPE_AGENT) + 1u;
        const int step = 4 * kp + (c >> 2), rt = (e & 3) * 64 + r0;
                char* dst = myws + WS_XS + (size_t)(step * 2 + (e >> 2)) * A_STAGE + (rt >> 4) * 1024 + ((c & 3) * 16 + (rt & 15)) * 16;
        asm volatile("" : "+v"(v[0]), "+v"(v[1]), "+v"(v[2]), "+v"(v[3]), "+v"(call));
#pragma unroll
        for (int i = 0; i < 4; ++i) store16_sys(dst + (i >> 1) * 2048 + (i & 1) * 4 * A_STAGE, v[i]);
        asm volatile("s_waitcnt vmcnt(0)\n\ts_barrier" ::: "memory");
        if (tid == 64) {
            const unsigned foff = (unsigned)WS_FLAG + (unsigned)(P.rank * NWG + bid) * 4u;
#pragma unroll
            for (int p = 0; p < NR; ++p) asm volatile("global_store_dword %0, %1, %2 sc0 sc1" :: "v"(foff), "v"(call), "s"(P.ws[p]) : "memory");
        }
    } else {
        call = __hip_atomic_load(ctr, __ATOMIC_RELAXED, __HIP_MEMORY_SCOPE_AGENT) + 1u;
        asm volatile("" : "+v"(call));
        const unsigned xcc = (unsigned)__builtin_amdgcn_s_getreg((3 << 11) | 20);
        if (wid == 4) {
            const unsigned ucall = __builtin_amdgcn_readfirstlane(call);
            unsigned t = ((unsigned)bid & ~7u) | (xcc & 7u);
#pragma unroll 1
            for (int i = 0; i < NWG; ++i) {
                unsigned old = 0;
                if (lane == 0) old = __hip_atomic_exchange((unsigned*)(myws + WS_CLAIM) + t * 16, ucall, __ATOMIC_RELAXED, __HIP_MEMORY_SCOPE_AGENT);
                if (__builtin_amdgcn_readfirstlane(old) != ucall) break;
                t = (t + 1) & (NWG - 1);
            }
            if (lane == 0) red[32] = t;
        }
        asm volatile("s_waitcnt lgkmcnt(0)\n\ts_barrier" ::: "memory");
        if (wid == 7 && lane == 0)
            xold = __hip_atomic_exchange((unsigned*)(myws + WS_XCC) + xcc * 32, call, __ATOMIC_RELAXED, __HIP_MEMORY_SCOPE_AGENT);
    }
    call = __builtin_amdgcn_readfirstlane(call);
    STAMP(1);

    const int tile = __builtin_amdgcn_readfirstlane(*(const volatile LAS unsigned*)(red + 32));
    const int srcr = AG_SRC(tile, P.map);
    const int tix = tile >> 3, half = tix & 1, n0 = (tix >> 1) * 128;

    const unsigned lds0 = (unsigned)(size_t)lds;
    int ready = 0;
    const char* const fp = myws + WS_FLAG + (size_t)srcr * (NWG * 4);
    auto ready_of = [&](u32x4 f) {
        const unsigned miss = ~(unsigned)__ballot(f[0] >= call && f[1] >= call && f[2] >= call && f[3] >= call);
        return miss ? 4 * (int)__builtin_ctz(miss) : NSTEP;
    };
    auto wait_ready = [&](int need, int slices) {
        const unsigned long long t0 = __builtin_amdgcn_s_memrealtime();
        unsigned polls = 0;
        for (;;) {
            ready = ready_of(load16_sys(fp + (lane & (slices - 1)) * 32 + half * 16));
            if (ready > 4 * slices) ready = 4 * slices;
            if (ready >= need) break;
            if (give_up(tmo, call, t0, polls)) { ready = NSTEP; break; }
        }
    };

    const char* gA = P.ws[srcr] + WS_XS + (size_t)half * A_STAGE + wid * 2048;
    const char* gB = P.w + n0 * 2;
    unsigned voffA[NSTAGE], voffB[NSTAGE];
    {
        const int row = 8 * (wid >> 1) + ((lane >> 2) & 7), chh = 2 * (wid & 1) + (lane >> 5);
        const int ch = chh * 4 + ((lane & 3) ^ ((row >> 2) & 3) ^ (chh & 1));
#pragma unroll
        for (int k = 0; k < NSTAGE; ++k) {
            voffA[k] = (unsigned)(lane * 16 + k * 2 * A_STAGE);
            voffB[k] = (unsigned)(row * (NL * 2) + ch * 16 + k * BK * NL * 2);
        }
    }
    const unsigned ldsWA = lds0 + LDS_A + wid * 2048, ldsWB = lds0 + LDS_B + wid * 1024;
    auto issue = [&](auto K, auto SLOT) {
        stage_dma<decltype(SLOT)::value>(ldsWA, ldsWB, voffA[decltype(K)::value], gA, voffB[decltype(K)::value], gB);
    };

    const int wm = wid >> 1, wc = wid & 1;
    const int g = lane >> 4, q4 = (lane & 15) >> 2, p4 = lane & 3;
    const unsigned rdA = LDS_A + wm * 4096 + lane * 16;
    const unsigned rdB0 = LDS_B + 2048 * g + 512 * (2 * wc + (p4 >> 1)) + 64 * q4 + 32 * ((p4 ^ g) & 1);
    unsigned rdB[2][2];
#pragma unroll
    for (int u = 0; u < 2; ++u)
#pragma unroll
        for (int v = 0; v < 2; ++v) rdB[u][v] = rdB0 + 16 * (u ^ (p4 >> 1)) + 8 * (v ^ (p4 & 1));

    f32x4 acc[4][4];
#pragma unroll
    for (int i = 0; i < 4; ++i)
#pragma unroll
        for (int j = 0; j < 4; ++j) acc[i][j] = (f32x4){0.f, 0.f, 0.f, 0.f};
    struct Frags { s16x8 x[4], w[4]; };
    Frags f0, f1;
#pragma unroll
    for (int j = 0; j < 4; ++j) { f1.x[j] = (s16x8){0, 0, 0, 0, 0, 0, 0, 0}; f1.w[j] = f1.x[j]; }
    auto frag = [&](Frags& f, int slot, int j) {
        const s16x4 lo = __builtin_amdgcn_ds_read_tr16_b64_v4i16((LAS s16x4*)(lds + rdB[j >> 1][j & 1] + slot * B_STAGE));
        const s16x4 hi = __builtin_amdgcn_ds_read_tr16_b64_v4i16((LAS s16x4*)(lds + rdB[(j >> 1) ^ 1][j & 1] + slot * B_STAGE + 256));
        f.w[j] = __builtin_shufflevector(lo, hi, 0, 1, 2, 3, 4, 5, 6, 7);
        f.x[j] = *(const LAS s16x8*)(lds + rdA + slot * A_STAGE + j * 1024);
    };
    auto mm = [&](const Frags& f, int i, int j) { acc[i][j] = __builtin_amdgcn_mfma_f32_16x16x32_bf16(f.w[j], f.x[i], acc[i][j], 0, 0, 0); };
#define SB __builtin_amdgcn_sched_barrier(0)
    auto step = [&](auto INFLIGHT, auto SLOT, auto K, Frags& cur, Frags& prev) {
        constexpr int slot = decltype(SLOT)::value, k = decltype(K)::value, ns = (slot + NSTAGE - 1) % NSTAGE;
        asm volatile("s_waitcnt vmcnt(%0) lgkmcnt(0)\n\ts_barrier" :: "n"(3 * decltype(INFLIGHT)::value) : "memory");
        frag(cur, slot, 0); SB; mm(prev, 0, 3); mm(prev, 1, 3); SB;
        frag(cur, slot, 1); SB; mm(prev, 2, 3); mm(prev, 3, 0); SB;
        if constexpr (k >= 0) { if (wc == 0) issue(IC<k>{}, IC<ns>{}); SB; }
        frag(cur, slot, 2); SB; mm(prev, 3, 1); mm(prev, 3, 2); SB;
        frag(cur, slot, 3); SB; mm(prev, 3, 3); SB;
        mm(cur, 0, 0); mm(cur, 0, 1); mm(cur, 1, 0); mm(cur, 1, 1); SB;
        mm(cur, 0, 2); mm(cur, 1, 2); mm(cur, 2, 0); mm(cur, 2, 1); mm(cur, 2, 2); SB;
        if constexpr (k >= 0) { if (wc != 0) issue(IC<k>{}, IC<ns>{}); SB; }
    };

    asm volatile("" :: "s"(gA), "s"(gB), "v"(voffB[0]));
    if (wid == 0) wait_ready(8, 4);
    asm volatile("s_barrier" ::: "memory");
    STAMP(2);
    asm volatile("s_nop 4" ::: "memory");
    issue(IC<0>{}, IC<0>{}); issue(IC<1>{}, IC<1>{}); issue(IC<2>{}, IC<2>{});
    gA += 3 * 2 * A_STAGE; gB += 3 * BK * NL * 2;
    for (int s = 0; s < NSTEP - NSTAGE; s += NSTAGE) {
        if (wid == 0 && ready < NSTEP && s > 0) {
            const int need = s + 8 < NSTEP ? s + 8 : NSTEP;
            ready = ready_of(*(const volatile LAS u32x4*)(lds + LDS_X + 256 + (lane & 31) * 32 + half * 16));
            if (ready < need) wait_ready(need, 32);
        }
        step(IC<2>{}, IC<0>{}, IC<0>{}, f0, f1);
        if (wid == 0 && ready < NSTEP) dma16_sys(lds0 + LDS_X + 256, fp + lane * 16);
        step(IC<2>{}, IC<1>{}, IC<1>{}, f1, f0);
        step(IC<2>{}, IC<2>{}, IC<2>{}, f0, f1);
        step(IC<2>{}, IC<3>{}, IC<3>{}, f1, f0);
        gA += NSTAGE * 2 * A_STAGE; gB += NSTAGE * BK * NL * 2;
    }
    step(IC<2>{}, IC<0>{}, IC<0>{}, f0, f1);
    step(IC<2>{}, IC<1>{}, IC<-1>{}, f1, f0);
    step(IC<1>{}, IC<2>{}, IC<-1>{}, f0, f1);
    step(IC<0>{}, IC<3>{}, IC<-1>{}, f1, f0);
    mm(f1, 0, 3); mm(f1, 1, 3); mm(f1, 2, 3); mm(f1, 3, 0); mm(f1, 3, 1); mm(f1, 3, 2); mm(f1, 3, 3);
#undef SB
    STAMP(3);

    float mx = 0.f;
#pragma unroll
    for (int i = 0; i < 4; ++i)
#pragma unroll
        for (int j = 0; j < 4; ++j)
#pragma unroll
            for (int r = 0; r < 4; ++r) mx = fmaxf(mx, fabsf(acc[i][j][r]));
    {
        const unsigned m = wave_umax(__float_as_uint(mx));
        if (lane == 0) red[wid] = m;
    }
    asm volatile("s_waitcnt lgkmcnt(0)\n\ts_barrier" ::: "memory");
    const unsigned par = call & 1u;
    if (wid == 0) {
        const u32x4 a = *(const LAS u32x4*)red, b = *(const LAS u32x4*)(red + 4);
        const unsigned m = umax(umax(umax(a[0], a[1]), umax(a[2], a[3])), umax(umax(b[0], b[1]), umax(b[2], b[3])));
        if (lane == 0)
            __hip_atomic_store((unsigned long long*)(myws + WS_LMAX + par * (NWG * 8)) + bid, ((unsigned long long)call << 32) | m,
                               __ATOMIC_RELAXED, __HIP_MEMORY_SCOPE_AGENT);
    }
    STAMP(4);
    f32x4 qv[4][4];
    const bool hi8 = (lane & 8) != 0;
    {
        f32x4 own[2][4], oth[2][4];
        LAS char* xw = lds + wid * 8192 + lane * 16;
        if (wc == 0) {
#pragma unroll
            for (int i = 0; i < 2; ++i)
#pragma unroll
                for (int j = 0; j < 4; ++j) { *(LAS f32x4*)(xw + (i * 4 + j) * 1024) = acc[2 + i][j]; own[i][j] = acc[i][j]; }
        } else {
#pragma unroll
            for (int i = 0; i < 2; ++i)
#pragma unroll
                for (int j = 0; j < 4; ++j) { *(LAS f32x4*)(xw + (i * 4 + j) * 1024) = acc[i][j]; own[i][j] = acc[2 + i][j]; }
        }
        asm volatile("s_waitcnt lgkmcnt(0)\n\ts_barrier" ::: "memory");
        const LAS char* xr = lds + (wid ^ 1) * 8192 + (lane ^ 8) * 16;
#pragma unroll
        for (int i = 0; i < 2; ++i)
#pragma unroll
            for (int j = 0; j < 4; ++j) oth[i][j] = *(const LAS f32x4*)(xr + (i * 4 + j) * 1024);
        const bool odd = g & 1;
#pragma unroll
        for (int i = 0; i < 2; ++i)
#pragma unroll
            for (int j = 0; j < 4; ++j) {
                const f32x4 a0 = own[i][j], a1 = own[i][j ^ 1], b0 = oth[i][j], b1 = oth[i][j ^ 1];
#pragma unroll
                for (int r = 0; r < 4; ++r) {
                    const float a = odd ? a1[r] : a0[r], b = odd ? b1[r] : b0[r];
                    qv[2 * i][j][r] = hi8 ? b : a;
                    qv[2 * i + 1][j][r] = hi8 ? a : b;
                }
                asm volatile("" : "+v"(qv[2 * i][j]), "+v"(qv[2 * i + 1][j]));
            }
    }
    char* op = P.out + (size_t)(srcr * MS + half * 256 + wm * 64 + wc * 32 + (lane & 7)) * NL + n0 + g * 16;
    const int col0 = hi8 ? 64 - wc * 64 : wc * 64, col1 = 64 - col0;
    asm volatile("" : "+v"(op));
    if (wid == 1 && srcr == P.rank && tix < NRED) {
        char* const dst = P.ws[lane & 7] + WS_GMAX + par * (NREP * 128) + (lane >> 3) * 128 + P.rank * 8;
        const char* gp = myws + WS_LMAX + par * (NWG * 8) + lane * 16;
        const unsigned long long t0 = __builtin_amdgcn_s_memrealtime();
        unsigned polls = 0;
        for (;;) {
            u32x4 a, b;
            load16x2_sys(gp, a, b);
            if (__all(a[1] == call && a[3] == call && b[1] == call && b[3] == call)) {
                const unsigned m = wave_umax(umax(umax(a[0], a[2]), umax(b[0], b[2])));
                __hip_atomic_store((unsigned long long*)dst, ((unsigned long long)call << 32) | m, __ATOMIC_RELAXED, __HIP_MEMORY_SCOPE_SYSTEM);
                break;
            }
            if (give_up(tmo, call, t0, polls)) break;
        }
    }
    if (wid == 2) {
        const char* gp = lane < 8 ? myws + WS_GMAX + par * (NREP * 128) + (tile & (NREP - 1)) * 128 + lane * 8 : (const char*)tmo;
        const unsigned long long t0 = __builtin_amdgcn_s_memrealtime();
        unsigned polls = 0, m = 0;
        bool dead = false;
        for (;;) {
            const u32x2 a = load8_sys(gp);
            if (__builtin_amdgcn_readlane(a[0], 8) == call) { dead = true; break; }
            if (__all(lane >= 8 || a[1] == call)) { m = a[0]; break; }
            if (give_up(tmo, call, t0, polls)) { dead = true; break; }
            __builtin_amdgcn_s_sleep(2);
        }
        m = umax(m, (unsigned)__builtin_amdgcn_update_dpp(0, (int)m, 0xB1, 0xf, 0xf, true));
        m = umax(m, (unsigned)__builtin_amdgcn_update_dpp(0, (int)m, 0x4E, 0xf, 0xf, true));
        m = umax(m, (unsigned)__builtin_amdgcn_update_dpp(0, (int)m, 0x141, 0xf, 0xf, true));
        const float amax = __uint_as_float(m);
        const float inv = amax > 0.f ? 127.0f / amax : 0.f;
        if (lane == 0) { red[16] = __float_as_uint(inv); red[17] = dead ? 1u : 0u; }
    }
    asm volatile("s_waitcnt lgkmcnt(0)\n\ts_barrier" ::: "memory");
    const float inv = __uint_as_float(red[16]);
    const unsigned anydead = red[17];
    STAMP(5);

#pragma unroll
    for (int i = 0; i < 4; ++i) {
        u32x4 o;
#pragma unroll
        for (int j = 0; j < 4; ++j) {
            unsigned b[4];
#pragma unroll
            for (int r = 0; r < 4; ++r) {
                b[r] = __float_as_uint(__builtin_fmaf(qv[i][j][r], inv, 12582912.0f));
            }
            const unsigned lo = __builtin_amdgcn_perm(b[1], b[0], 0x0c0c0400u), hi = __builtin_amdgcn_perm(b[3], b[2], 0x04000c0cu);
            o[j] = lo | hi;
        }
        if (anydead) o = (u32x4){0x80808080u, 0x80808080u, 0x80808080u, 0x80808080u};
        store16_sys(op + (size_t)(i * 8) * NL + ((i & 1) ? col1 : col0), o);
        __builtin_amdgcn_sched_barrier(0);
    }
    if (tid == 0) __hip_atomic_store(ctr, call, __ATOMIC_RELAXED, __HIP_MEMORY_SCOPE_AGENT);
    if (wid == 7 && __builtin_amdgcn_readfirstlane(xold) != call) asm volatile("buffer_inv sc0 sc1" ::: "memory");
    STAMP(6);
}

}

extern "C" void kernel_launch(int rank, int nranks, ncclComm_t comm,
                              hipStream_t stream,
                              void* const* d_in, const int* in_sizes, int n_in,
                              void* d_out, int out_size,
                              const int* in_dtypes, int out_dtype,
                              void* const* d_out_all,
                              void* const* d_ws_all, size_t ws_bytes) {
    (void)comm; (void)in_dtypes; (void)out_dtype; (void)d_out_all;
    if (nranks != NR || n_in < 2 || rank < 0 || rank >= NR || in_sizes[0] != MS * KD || in_sizes[1] != KD * NL ||
        out_size != NR * MS * NL || ws_bytes < WS_END) {
        fprintf(stderr, "kernel_launch: unexpected problem shape\n");
        return;
    }
    Params P;
    P.x = (const char*)d_in[0];
    P.w = (const char*)d_in[1];
    P.out = (char*)d_out;
    for (int p = 0; p < NR; ++p) P.ws[p] = (char*)d_ws_all[p];
    P.myws = P.ws[rank];
    P.rank = rank;
    P.map = AG_MAP(rank);
    HIP_CHECK(hipFuncSetAttribute((const void*)ag_gemm_q8, hipFuncAttributeMaxDynamicSharedMemorySize, LDS_BYTES));
    hipLaunchKernelGGL(ag_gemm_q8, dim3(NWG), dim3(512), LDS_BYTES, stream, P);
    HIP_CHECK(hipGetLastError());
}
```

```cpp
#include <hip/hip_runtime.h>
#include <rccl/rccl.h>
#include <cstdio>

#define HIP_CHECK(expr) do { hipError_t _e = (expr); if (_e != hipSuccess) { \
    fprintf(stderr, "HIP error %s at %s:%d\n", hipGetErrorString(_e), __FILE__, __LINE__); \
    return; } } while (0)

namespace {

#define AG_MAP(rank) ((((0x4Du >> (rank)) & 1u) != 0u) ? 0x75416320u : 0x63207541u)
#define AG_SRC(g, map) ((int)(((map) >> (4 * ((g) & 7))) & 7u))

#define LAS __attribute__((address_space(3)))
typedef short s16x4 __attribute__((ext_vector_type(4)));
typedef short s16x8 __attribute__((ext_vector_type(8)));
typedef float f32x4 __attribute__((ext_vector_type(4)));
typedef unsigned u32x4 __attribute__((ext_vector_type(4)));
typedef unsigned u32x2 __attribute__((ext_vector_type(2)));

constexpr int NR = 8;
constexpr int MS = 512;
constexpr int KD = 4096;
constexpr int NL = 2048;
constexpr int NWG = 256;
constexpr int BK = 32, NSTEP = KD / BK;
constexpr int NSTAGE = 4;
constexpr int A_STAGE = 256 * BK * 2;
constexpr int B_STAGE = BK * 128 * 2;
constexpr int LDS_A = 0, LDS_B = NSTAGE * A_STAGE, LDS_X = LDS_B + NSTAGE * B_STAGE;
constexpr int LDS_BYTES = LDS_X + 256 + 1024;
constexpr int NRED = 2;
constexpr int NREP = 8;

constexpr size_t WS_XS = 0;
constexpr size_t WS_FLAG = (size_t)MS * KD * 2;
constexpr size_t WS_LMAX = WS_FLAG + 65536;
constexpr size_t WS_GMAX = WS_LMAX + 65536;
constexpr size_t WS_CTR = WS_GMAX + 65536;
constexpr size_t WS_TMO = WS_CTR + 4096;
constexpr size_t WS_XCC = WS_TMO + 4096;
constexpr size_t WS_CLAIM = WS_XCC + 4096;
constexpr size_t WS_DBG = WS_CLAIM + 16384;
constexpr size_t WS_END = WS_DBG + 65536;
constexpr unsigned long long DEADLINE = 50000000ull;

template <int N> struct IC { static constexpr int value = N; };

struct Params {
    const char* x;
    const char* w;
    char* out;
    char* myws;
    char* ws[NR];
    int rank;
    unsigned map;
};

__device__ __forceinline__ u32x2 load8_sys(const char* p) {
    u32x2 v;
    asm volatile("global_load_dwordx2 %0, %1, off sc0 sc1\n\ts_waitcnt vmcnt(0)" : "=&v"(v) : "v"(p) : "memory");
    return v;
}
__device__ __forceinline__ u32x4 load16_sys(const char* p) {
    u32x4 v;
    asm volatile("global_load_dwordx4 %0, %1, off sc0 sc1\n\ts_waitcnt vmcnt(0)" : "=&v"(v) : "v"(p) : "memory");
    return v;
}
__device__ __forceinline__ void load16x2_sys(const char* p, u32x4& a, u32x4& b) {
    asm volatile("global_load_dwordx4 %0, %2, off sc0 sc1\n\tglobal_load_dwordx4 %1, %2, off offset:1024 sc0 sc1\n\ts_waitcnt vmcnt(0)"
                 : "=&v"(a), "=&v"(b) : "v"(p) : "memory");
}
__device__ __forceinline__ void dma16_sys(unsigned lds_addr, const char* p) {
    unsigned keep;
    asm volatile("s_mov_b32 %0, m0\n\ts_mov_b32 m0, %1\n\ts_nop 0\n\tglobal_load_lds_dwordx4 %2, off sc0 sc1\n\ts_mov_b32 m0, %0"
                 : "=&s"(keep) : "s"(lds_addr), "v"(p) : "memory");
}
template <int SLOT>
__device__ __forceinline__ void stage_dma(unsigned ldsA, unsigned ldsB, unsigned va, const char* ga, unsigned vb, const char* gb) {
    asm volatile("s_add_i32 m0, %0, %6\n\ts_nop 0\n\t"
                 "global_load_lds_dwordx4 %2, %3\n\t"
                 "global_load_lds_dwordx4 %2, %3 offset:1024\n\t"
                 "s_add_i32 m0, %1, %7\n\ts_nop 0\n\t"
                 "global_load_lds_dwordx4 %4, %5"
                 :: "s"(ldsA), "s"(ldsB), "v"(va), "s"(ga), "v"(vb), "s"(gb), "n"(SLOT * A_STAGE), "n"(SLOT * B_STAGE) : "memory", "scc");
}
__device__ __forceinline__ void store16_sys(char* p, u32x4 v) {
    asm volatile("global_store_dwordx4 %0, %1, off sc0 sc1\n\ts_nop 1" :: "v"(p), "v"(v) : "memory");
}
__device__ __forceinline__ unsigned umax(unsigned a, unsigned b) { return a > b ? a : b; }
__device__ __forceinline__ unsigned wave_umax(unsigned v) {
    v = umax(v, (unsigned)__builtin_amdgcn_update_dpp(0, (int)v, 0xB1, 0xf, 0xf, true));
    v = umax(v, (unsigned)__builtin_amdgcn_update_dpp(0, (int)v, 0x4E, 0xf, 0xf, true));
    v = umax(v, (unsigned)__builtin_amdgcn_update_dpp(0, (int)v, 0x141, 0xf, 0xf, true));
    v = umax(v, (unsigned)__builtin_amdgcn_update_dpp(0, (int)v, 0x140, 0xf, 0xf, true));
    const unsigned a = __builtin_amdgcn_readlane(v, 0), b = __builtin_amdgcn_readlane(v, 16), c = __builtin_amdgcn_readlane(v, 32), d = __builtin_amdgcn_readlane(v, 48);
    return umax(umax(a, b), umax(c, d));
}
__device__ __forceinline__ bool give_up(unsigned* tmo, unsigned call, unsigned long long t0, unsigned& polls) {
    if ((++polls & 15u) != 0u) return false;
    const unsigned t = __builtin_amdgcn_readfirstlane(__hip_atomic_load(tmo, __ATOMIC_RELAXED, __HIP_MEMORY_SCOPE_AGENT));
    if (t == call || __builtin_amdgcn_s_memrealtime() - t0 > DEADLINE) {
        __hip_atomic_store(tmo, call, __ATOMIC_RELAXED, __HIP_MEMORY_SCOPE_AGENT);
        return true;
    }
    return false;
}

#define STAMP(i) do { } while (0)

__global__ __launch_bounds__(512, 2) void ag_gemm_q8(Params P) {
    extern __shared__ __attribute__((aligned(16))) char smem[];
    LAS char* lds = (LAS char*)smem;
    const int tid = threadIdx.x, lane = tid & 63, wid = __builtin_amdgcn_readfirstlane(tid >> 6);
    const int bid = blockIdx.x;
    char* const myws = P.myws;
    asm volatile("" :: "s"(P.x), "s"(P.w), "s"(P.out), "s"(P.myws), "s"(P.rank), "s"(P.map), "s"(P.ws[0]), "s"(P.ws[1]), "s"(P.ws[2]),
                 "s"(P.ws[3]), "s"(P.ws[4]), "s"(P.ws[5]), "s"(P.ws[6]), "s"(P.ws[7]));
    unsigned* const tmo = (unsigned*)(myws + WS_TMO);
    unsigned* const ctr = (unsigned*)(myws + WS_CTR) + bid;
    LAS unsigned* const red = (LAS unsigned*)(lds + LDS_X);
    STAMP(0);

    unsigned call;
    unsigned xold = 0;
    if (wid < 4) {
        const int kp = bid >> 3, e = bid & 7;
        const int r0 = tid >> 3, c = tid & 7;
        const char* src = P.x + (size_t)(e * 64 + r0) * (KD * 2) + kp * 256 + c * 16;
        if (kp >= 2) __builtin_amdgcn_s_sleep(5);
        u32x4 v[4];
#pragma unroll
        for (int i = 0; i < 4; ++i)
            v[i] = __builtin_nontemporal_load((const u32x4*)(src + (size_t)(i >> 1) * 32 * (KD * 2) + (i & 1) * 128));
        call = __hip_atomic_load(ctr, __ATOMIC_RELAXED, __HIP_MEMORY_SCOPE_AGENT) + 1u;
        const int step = 4 * kp + (c >> 2), rt = (e & 3) * 64 + r0;
        char* dst = myws + WS_XS + (size_t)(step * 2 + (e >> 2)) * A_STAGE + (rt >> 4) * 1024 + ((c & 3) * 16 + (rt & 15)) * 16;
        asm volatile("" : "+v"(v[0]), "+v"(v[1]), "+v"(v[2]), "+v"(v[3]), "+v"(call));
#pragma unroll
        for (int i = 0; i < 4; ++i) store16_sys(dst + (i >> 1) * 2048 + (i & 1) * 4 * A_STAGE, v[i]);
        asm volatile("s_waitcnt vmcnt(0)\n\ts_barrier" ::: "memory");
        if (tid == 64) {
            const unsigned foff = (unsigned)WS_FLAG + (unsigned)(P.rank * NWG + bid) * 4u;
#pragma unroll
            for (int p = 0; p < NR; ++p) asm volatile("global_store_dword %0, %1, %2 sc0 sc1" :: "v"(foff), "v"(call), "s"(P.ws[p]) : "memory");
        }
    } else {
        call = __hip_atomic_load(ctr, __ATOMIC_RELAXED, __HIP_MEMORY_SCOPE_AGENT) + 1u;
        asm volatile("" : "+v"(call));
        const unsigned xcc = (unsigned)__builtin_amdgcn_s_getreg((3 << 11) | 20);
        if (wid == 4) {
            const unsigned ucall = __builtin_amdgcn_readfirstlane(call);
            unsigned t = ((unsigned)bid & ~7u) | (xcc & 7u);
#pragma unroll 1
            for (int i = 0; i < NWG; ++i) {
                unsigned old = 0;
                if (lane == 0) old = __hip_atomic_exchange((unsigned*)(myws + WS_CLAIM) + t * 16, ucall, __ATOMIC_RELAXED, __HIP_MEMORY_SCOPE_AGENT);
                if (__builtin_amdgcn_readfirstlane(old) != ucall) break;
                t = (t + 1) & (NWG - 1);
            }
            if (lane == 0) red[32] = t;
        }
        asm volatile("s_waitcnt lgkmcnt(0)\n\ts_barrier" ::: "memory");
        if (wid == 7 && lane == 0)
            xold = __hip_atomic_exchange((unsigned*)(myws + WS_XCC) + xcc * 32, call, __ATOMIC_RELAXED, __HIP_MEMORY_SCOPE_AGENT);
    }
    call = __builtin_amdgcn_readfirstlane(call);
    STAMP(1);

    const int tile = __builtin_amdgcn_readfirstlane(*(const volatile LAS unsigned*)(red + 32));
    const int srcr = AG_SRC(tile, P.map);
    const int tix = tile >> 3, half = tix & 1, n0 = (tix >> 1) * 128;

    const unsigned lds0 = (unsigned)(size_t)lds;
    int ready = 0;
    const char* const fp = myws + WS_FLAG + (size_t)srcr * (NWG * 4);
    auto ready_of = [&](u32x4 f) {
        const unsigned miss = ~(unsigned)__ballot(f[0] >= call && f[1] >= call && f[2] >= call && f[3] >= call);
        return miss ? 4 * (int)__builtin_ctz(miss) : NSTEP;
    };
    auto wait_ready = [&](int need, int slices) {
        const unsigned long long t0 = __builtin_amdgcn_s_memrealtime();
        unsigned polls = 0;
        for (;;) {
            ready = ready_of(load16_sys(fp + (lane & (slices - 1)) * 32 + half * 16));
            if (ready > 4 * slices) ready = 4 * slices;
            if (ready >= need) break;
            if (give_up(tmo, call, t0, polls)) { ready = NSTEP; break; }
        }
    };

    const char* gA = P.ws[srcr] + WS_XS + (size_t)half * A_STAGE + wid * 2048;
    const char* gB = P.w + n0 * 2;
    unsigned voffA[NSTAGE], voffB[NSTAGE];
    {
        const int row = 8 * (wid >> 1) + ((lane >> 2) & 7), chh = 2 * (wid & 1) + (lane >> 5);
        const int ch = chh * 4 + ((lane & 3) ^ ((row >> 2) & 3) ^ (chh & 1));
#pragma unroll
        for (int k = 0; k < NSTAGE; ++k) {
            voffA[k] = (unsigned)(lane * 16 + k * 2 * A_STAGE);
            voffB[k] = (unsigned)(row * (NL * 2) + ch * 16 + k * BK * NL * 2);
        }
    }
    const unsigned ldsWA = lds0 + LDS_A + wid * 2048, ldsWB = lds0 + LDS_B + wid * 1024;
    auto issue = [&](auto K, auto SLOT) {
        stage_dma<decltype(SLOT)::value>(ldsWA, ldsWB, voffA[decltype(K)::value], gA, voffB[decltype(K)::value], gB);
    };

    const int wm = wid >> 1, wc = wid & 1;
    const int g = lane >> 4, q4 = (lane & 15) >> 2, p4 = lane & 3;
    const unsigned rdA = LDS_A + wm * 4096 + lane * 16;
    const unsigned rdB0 = LDS_B + 2048 * g + 512 * (2 * wc + (p4 >> 1)) + 64 * q4 + 32 * ((p4 ^ g) & 1);
    unsigned rdB[2][2];
#pragma unroll
    for (int u = 0; u < 2; ++u)
#pragma unroll
        for (int v = 0; v < 2; ++v) rdB[u][v] = rdB0 + 16 * (u ^ (p4 >> 1)) + 8 * (v ^ (p4 & 1));

    f32x4 acc[4][4];
#pragma unroll
    for (int i = 0; i < 4; ++i)
#pragma unroll
        for (int j = 0; j < 4; ++j) acc[i][j] = (f32x4){0.f, 0.f, 0.f, 0.f};
    struct Frags { s16x8 x[4], w[4]; };
    Frags f0, f1;
#pragma unroll
    for (int j = 0; j < 4; ++j) { f1.x[j] = (s16x8){0, 0, 0, 0, 0, 0, 0, 0}; f1.w[j] = f1.x[j]; }
    auto frag = [&](Frags& f, int slot, int j) {
        const s16x4 lo = __builtin_amdgcn_ds_read_tr16_b64_v4i16((LAS s16x4*)(lds + rdB[j >> 1][j & 1] + slot * B_STAGE));
        const s16x4 hi = __builtin_amdgcn_ds_read_tr16_b64_v4i16((LAS s16x4*)(lds + rdB[(j >> 1) ^ 1][j & 1] + slot * B_STAGE + 256));
        f.w[j] = __builtin_shufflevector(lo, hi, 0, 1, 2, 3, 4, 5, 6, 7);
        f.x[j] = *(const LAS s16x8*)(lds + rdA + slot * A_STAGE + j * 1024);
    };
    auto mm = [&](const Frags& f, int i, int j) { acc[i][j] = __builtin_amdgcn_mfma_f32_16x16x32_bf16(f.w[j], f.x[i], acc[i][j], 0, 0, 0); };
#define SB __builtin_amdgcn_sched_barrier(0)
    auto step = [&](auto INFLIGHT, auto SLOT, auto K, Frags& cur, Frags& prev) {
        constexpr int slot = decltype(SLOT)::value, k = decltype(K)::value, ns = (slot + NSTAGE - 1) % NSTAGE;
        asm volatile("s_waitcnt vmcnt(%0) lgkmcnt(0)\n\ts_barrier" :: "n"(3 * decltype(INFLIGHT)::value) : "memory");
        frag(cur, slot, 0); SB; mm(prev, 0, 3); mm(prev, 1, 3); SB;
        frag(cur, slot, 1); SB; mm(prev, 2, 3); mm(prev, 3, 0); SB;
        if constexpr (k >= 0) { if (wc == 0) issue(IC<k>{}, IC<ns>{}); SB; }
        frag(cur, slot, 2); SB; mm(prev, 3, 1); mm(prev, 3, 2); SB;
        frag(cur, slot, 3); SB; mm(prev, 3, 3); SB;
        mm(cur, 0, 0); mm(cur, 0, 1); mm(cur, 1, 0); mm(cur, 1, 1); SB;
        mm(cur, 0, 2); mm(cur, 1, 2); mm(cur, 2, 0); mm(cur, 2, 1); mm(cur, 2, 2); SB;
        if constexpr (k >= 0) { if (wc != 0) issue(IC<k>{}, IC<ns>{}); SB; }
    };

    asm volatile("" :: "s"(gA), "s"(gB), "v"(voffB[0]));
    if (wid != 0) {
#pragma unroll
        for (int k = 0; k < NSTAGE - 1; ++k)
            asm volatile("s_mov_b32 m0, %0\n\ts_nop 4\n\tglobal_load_lds_dwordx4 %1, %2" :: "s"(ldsWB + k * B_STAGE), "v"(voffB[k]), "s"(gB) : "memory");
    }
    if (wid == 0) wait_ready(8, 4);
    asm volatile("s_barrier" ::: "memory");
    STAMP(2);
    asm volatile("s_nop 4" ::: "memory");
    issue(IC<0>{}, IC<0>{}); issue(IC<1>{}, IC<1>{}); issue(IC<2>{}, IC<2>{});
    gA += 3 * 2 * A_STAGE; gB += 3 * BK * NL * 2;
    for (int s = 0; s < NSTEP - NSTAGE; s += NSTAGE) {
        if (wid == 0 && ready < NSTEP && s > 0) {
            const int need = s + 8 < NSTEP ? s + 8 : NSTEP;
            ready = ready_of(*(const volatile LAS u32x4*)(lds + LDS_X + 256 + (lane & 31) * 32 + half * 16));
            if (ready < need) wait_ready(need, 32);
        }
        step(IC<2>{}, IC<0>{}, IC<0>{}, f0, f1);
        if (wid == 0 && ready < NSTEP) dma16_sys(lds0 + LDS_X + 256, fp + lane * 16);
        step(IC<2>{}, IC<1>{}, IC<1>{}, f1, f0);
        step(IC<2>{}, IC<2>{}, IC<2>{}, f0, f1);
        step(IC<2>{}, IC<3>{}, IC<3>{}, f1, f0);
        gA += NSTAGE * 2 * A_STAGE; gB += NSTAGE * BK * NL * 2;
    }
    step(IC<2>{}, IC<0>{}, IC<0>{}, f0, f1);
    step(IC<2>{}, IC<1>{}, IC<-1>{}, f1, f0);
    step(IC<1>{}, IC<2>{}, IC<-1>{}, f0, f1);
    step(IC<0>{}, IC<3>{}, IC<-1>{}, f1, f0);
    mm(f1, 0, 3); mm(f1, 1, 3); mm(f1, 2, 3); mm(f1, 3, 0); mm(f1, 3, 1); mm(f1, 3, 2); mm(f1, 3, 3);
#undef SB
    STAMP(3);

    float mx = 0.f;
#pragma unroll
    for (int i = 0; i < 4; ++i)
#pragma unroll
        for (int j = 0; j < 4; ++j)
#pragma unroll
            for (int r = 0; r < 4; ++r) mx = fmaxf(mx, fabsf(acc[i][j][r]));
    {
        const unsigned m = wave_umax(__float_as_uint(mx));
        if (lane == 0) red[wid] = m;
    }
    asm volatile("s_waitcnt lgkmcnt(0)\n\ts_barrier" ::: "memory");
    const unsigned par = call & 1u;
    if (wid == 0) {
        const u32x4 a = *(const LAS u32x4*)red, b = *(const LAS u32x4*)(red + 4);
        const unsigned m = umax(umax(umax(a[0], a[1]), umax(a[2], a[3])), umax(umax(b[0], b[1]), umax(b[2], b[3])));
        if (lane == 0)
            __hip_atomic_store((unsigned long long*)(myws + WS_LMAX + par * (NWG * 8)) + bid, ((unsigned long long)call << 32) | m,
                               __ATOMIC_RELAXED, __HIP_MEMORY_SCOPE_AGENT);
    }
    STAMP(4);
    f32x4 qv[4][4];
    const bool hi8 = (lane & 8) != 0;
    {
        f32x4 own[2][4], oth[2][4];
        LAS char* xw = lds + wid * 8192 + lane * 16;
        if (wc == 0) {
#pragma unroll
            for (int i = 0; i < 2; ++i)
#pragma unroll
                for (int j = 0; j < 4; ++j) { *(LAS f32x4*)(xw + (i * 4 + j) * 1024) = acc[2 + i][j]; own[i][j] = acc[i][j]; }
        } else {
#pragma unroll
            for (int i = 0; i < 2; ++i)
#pragma unroll
                for (int j = 0; j < 4; ++j) { *(LAS f32x4*)(xw + (i * 4 + j) * 1024) = acc[i][j]; own[i][j] = acc[2 + i][j]; }
        }
        asm volatile("s_waitcnt lgkmcnt(0)\n\ts_barrier" ::: "memory");
        const LAS char* xr = lds + (wid ^ 1) * 8192 + (lane ^ 8) * 16;
#pragma unroll
        for (int i = 0; i < 2; ++i)
#pragma unroll
            for (int j = 0; j < 4; ++j) oth[i][j] = *(const LAS f32x4*)(xr + (i * 4 + j) * 1024);
        const bool odd = g & 1;
#pragma unroll
        for (int i = 0; i < 2; ++i)
#pragma unroll
            for (int j = 0; j < 4; ++j) {
                const f32x4 a0 = own[i][j], a1 = own[i][j ^ 1], b0 = oth[i][j], b1 = oth[i][j ^ 1];
#pragma unroll
                for (int r = 0; r < 4; ++r) {
                    const float a = odd ? a1[r] : a0[r], b = odd ? b1[r] : b0[r];
                    qv[2 * i][j][r] = hi8 ? b : a;
                    qv[2 * i + 1][j][r] = hi8 ? a : b;
                }
                asm volatile("" : "+v"(qv[2 * i][j]), "+v"(qv[2 * i + 1][j]));
            }
    }
    char* op = P.out + (size_t)(srcr * MS + half * 256 + wm * 64 + wc * 32 + (lane & 7)) * NL + n0 + g * 16;
    const int col0 = hi8 ? 64 - wc * 64 : wc * 64, col1 = 64 - col0;
    asm volatile("" : "+v"(op));
    if (wid == 1 && srcr == P.rank && tix < NRED) {
        char* const dst = P.ws[lane & 7] + WS_GMAX + par * (NREP * 128) + (lane >> 3) * 128 + P.rank * 8;
        const char* gp = myws + WS_LMAX + par * (NWG * 8) + lane * 16;
        const unsigned long long t0 = __builtin_amdgcn_s_memrealtime();
        unsigned polls = 0;
        for (;;) {
            u32x4 a, b;
            load16x2_sys(gp, a, b);
            if (__all(a[1] == call && a[3] == call && b[1] == call && b[3] == call)) {
                const unsigned m = wave_umax(umax(umax(a[0], a[2]), umax(b[0], b[2])));
                __hip_atomic_store((unsigned long long*)dst, ((unsigned long long)call << 32) | m, __ATOMIC_RELAXED, __HIP_MEMORY_SCOPE_SYSTEM);
                break;
            }
            if (give_up(tmo, call, t0, polls)) break;
        }
    }
    if (wid == 2) {
        const char* gp = lane < 8 ? myws + WS_GMAX + par * (NREP * 128) + (tile & (NREP - 1)) * 128 + lane * 8 : (const char*)tmo;
        const unsigned long long t0 = __builtin_amdgcn_s_memrealtime();
        unsigned polls = 0, m = 0;
        bool dead = false;
        for (;;) {
            const u32x2 a = load8_sys(gp);
            if (__builtin_amdgcn_readlane(a[0], 8) == call) { dead = true; break; }
            if (__all(lane >= 8 || a[1] == call)) { m = a[0]; break; }
            if (give_up(tmo, call, t0, polls)) { dead = true; break; }
            __builtin_amdgcn_s_sleep(2);
        }
        m = umax(m, (unsigned)__builtin_amdgcn_update_dpp(0, (int)m, 0xB1, 0xf, 0xf, true));
        m = umax(m, (unsigned)__builtin_amdgcn_update_dpp(0, (int)m, 0x4E, 0xf, 0xf, true));
        m = umax(m, (unsigned)__builtin_amdgcn_update_dpp(0, (int)m, 0x141, 0xf, 0xf, true));
        const float amax = __uint_as_float(m);
        const float inv = amax > 0.f ? 127.0f / amax : 0.f;
        if (lane == 0) { red[16] = __float_as_uint(inv); red[17] = dead ? 1u : 0u; }
    }
    asm volatile("s_waitcnt lgkmcnt(0)\n\ts_barrier" ::: "memory");
    const float inv = __uint_as_float(red[16]);
    const unsigned anydead = red[17];
    STAMP(5);

#pragma unroll
    for (int i = 0; i < 4; ++i) {
        u32x4 o;
#pragma unroll
        for (int j = 0; j < 4; ++j) {
            unsigned b[4];
#pragma unroll
            for (int r = 0; r < 4; ++r) {
                b[r] = __float_as_uint(__builtin_fmaf(qv[i][j][r], inv, 12582912.0f));
            }
            const unsigned lo = __builtin_amdgcn_perm(b[1], b[0], 0x0c0c0400u), hi = __builtin_amdgcn_perm(b[3], b[2], 0x04000c0cu);
            o[j] = lo | hi;
        }
        if (anydead) o = (u32x4){0x80808080u, 0x80808080u, 0x80808080u, 0x80808080u};
        store16_sys(op + (size_t)(i * 8) * NL + ((i & 1) ? col1 : col0), o);
        __builtin_amdgcn_sched_barrier(0);
    }
    if (tid == 0) __hip_atomic_store(ctr, call, __ATOMIC_RELAXED, __HIP_MEMORY_SCOPE_AGENT);
    if (wid == 7 && __builtin_amdgcn_readfirstlane(xold) != call) asm volatile("buffer_inv sc0 sc1" ::: "memory");
    STAMP(6);
}

}

extern "C" void kernel_launch(int rank, int nranks, ncclComm_t comm,
                              hipStream_t stream,
                              void* const* d_in, const int* in_sizes, int n_in,
                              void* d_out, int out_size,
                              const int* in_dtypes, int out_dtype,
                              void* const* d_out_all,
                              void* const* d_ws_all, size_t ws_bytes) {
    (void)comm; (void)in_dtypes; (void)out_dtype; (void)d_out_all;
    if (nranks != NR || n_in < 2 || rank < 0 || rank >= NR || in_sizes[0] != MS * KD || in_sizes[1] != KD * NL ||
        out_size != NR * MS * NL || ws_bytes < WS_END) {
        fprintf(stderr, "kernel_launch: unexpected problem shape\n");
        return;
    }
    Params P;
    P.x = (const char*)d_in[0];
    P.w = (const char*)d_in[1];
    P.out = (char*)d_out;
    for (int p = 0; p < NR; ++p) P.ws[p] = (char*)d_ws_all[p];
    P.myws = P.ws[rank];
    P.rank = rank;
    P.map = AG_MAP(rank);
    HIP_CHECK(hipFuncSetAttribute((const void*)ag_gemm_q8, hipFuncAttributeMaxDynamicSharedMemorySize, LDS_BYTES));
    hipLaunchKernelGGL(ag_gemm_q8, dim3(NWG), dim3(512), LDS_BYTES, stream, P);
    HIP_CHECK(hipGetLastError());
}
```
